```python
import numpy as np

kernel_name = "gpu_ag_gemm_m2048_k2048_n2048_bf16_gelu"


ABSMAX_RTOL = 0.008

COMPUTE_DTYPE = "bfloat16"
EPILOGUE = "gelu"
IN_QUANT = None
OUT_QUANT = None
IN_DTYPES = {"x": "bfloat16", "w_mat": "bfloat16"}
WORLD_SIZE = 8

def _to_bf16(x):
    u = np.ascontiguousarray(x, dtype=np.float32).view(np.uint32)
    return ((u + 0x7FFF + ((u >> 16) & 1)) & 0xFFFF0000).view(np.float32)

def setup_inputs():

    x = _to_bf16(np.random.randn(2048, 2048).astype(np.float32))
    w_mat = _to_bf16((np.random.randn(2048, 2048) / np.sqrt(2048)).astype(np.float32))
    return {"x": x, "w_mat": w_mat}

def _epilogue(y):
    c = 0.7978845608028654
    return (0.5 * y * (1.0 + np.tanh(c * (y + 0.044715 * y ** 3)))).astype(np.float32)

def reference(x, w_mat):
    y = (x @ w_mat).astype(np.float32)
    return _epilogue(y)
```

```cpp
#include <hip/hip_runtime.h>
#include <rccl/rccl.h>
#include <cstdio>
#include <cstdint>

typedef unsigned u32x4 __attribute__((ext_vector_type(4)));
typedef unsigned u32x2 __attribute__((ext_vector_type(2)));
typedef short s16x4 __attribute__((ext_vector_type(4)));
typedef short s16x8 __attribute__((ext_vector_type(8)));
typedef __bf16 bf16x8 __attribute__((ext_vector_type(8)));
typedef __bf16 bf16x2 __attribute__((ext_vector_type(2)));
typedef float f32x4 __attribute__((ext_vector_type(4)));
typedef float f32x2 __attribute__((ext_vector_type(2)));
#define LDS_AS __attribute__((address_space(3)))
#define GLB_AS __attribute__((address_space(1)))

namespace {
constexpr int NR = 8;
constexpr int KD = 2048;
constexpr int NS = 256;
constexpr int MS = 256;
constexpr int NCH = 16;
constexpr size_t PKT = 4096;
constexpr size_t SRC_BYTES = 16 * NCH * PKT;
constexpr size_t PAR_BYTES = NR * SRC_BYTES;
constexpr size_t FLAG_OFF = 2 * PAR_BYTES;
constexpr size_t FLAG_PAR = (size_t)NR * 16 * NCH * 16;
constexpr size_t CTR_OFF = FLAG_OFF + 2 * FLAG_PAR;
constexpr size_t TMO_OFF = CTR_OFF + 256 * 64;
constexpr size_t STAMP_OFF = FLAG_OFF + 256 * 1024;
constexpr size_t WS_NEED = FLAG_OFF + 512 * 1024;
constexpr unsigned long long DEADLINE = 20000000ull;
constexpr int LDS_W = 2048 * 64;
constexpr int LDS_BYTES = LDS_W + 64;
#define LAG 2

struct Params {
  const char* x; const char* w; char* out; char* ws[NR]; int rank; int pad;
};

#define STAMP(i)

constexpr int after_cnt(int p) {
  int n = 0;
  if (p + LAG <= 7) { n = 4 * LAG; for (int i = p; i < p + LAG; ++i) if (i >= LAG) ++n; }
  else { n = 4 * (7 - p); for (int i = p; i <= 7; ++i) if (i >= LAG) ++n; n += p - (8 - LAG); }
  return n;
}
#define PSTAMP(i)

__device__ __forceinline__ float gelu(float y) {
  const float c2 = 2.0f * 0.7978845608028654f * 1.4426950408889634f;
  float u = y * (c2 + (c2 * 0.044715f) * y * y);
  float e = __builtin_amdgcn_exp2f(-u);
  return y * __builtin_amdgcn_rcpf(1.0f + e);
}

__global__ void __launch_bounds__(320, 2) ag_gemm(Params P) {
  extern __shared__ __attribute__((aligned(16))) char lds[];
  const int tid = threadIdx.x;
  const int lane = tid & 63;
  const int wave = __builtin_amdgcn_readfirstlane(tid >> 6);
  const int b = blockIdx.x, xcd = b & 7, bi = b >> 3;
  char* myws = P.ws[P.rank];
  unsigned* ctrp = (unsigned*)(myws + CTR_OFF + b * 64 + wave * 4);
  unsigned* tmo = (unsigned*)(myws + TMO_OFF);
  if (wave != 4) STAMP(0);

  if (wave == 4) {
    __builtin_amdgcn_s_barrier();
    if (bi >= 28) {
      unsigned c0 = __hip_atomic_load(ctrp, __ATOMIC_RELAXED, __HIP_MEMORY_SCOPE_SYSTEM);
      if (lane == 0) __hip_atomic_store(ctrp, c0 + 1, __ATOMIC_RELAXED, __HIP_MEMORY_SCOPE_SYSTEM);
      return;
    }
    unsigned cv;
    asm volatile("global_load_dword %0, %1, off sc0 sc1" : "=v"(cv) : "v"(ctrp) : "memory");
    const int pair = xcd * 4 + bi / 7, q = bi % 7;
    const int t = pair >> 1, h = pair & 1;
    const unsigned peer = (unsigned)(P.rank + 1 + q) & 7u;
    const char* src = P.x + (size_t)(t * 16 + (lane & 15)) * 4096 + h * 256 + (lane >> 4) * 16;
    u32x4 d[8][4];
#pragma unroll
    for (int j = 0; j < 8; ++j)
#pragma unroll
      for (int f = 0; f < 4; ++f)
        asm volatile("global_load_dwordx4 %0, %1, off offset:%c2" : "=v"(d[j][f]) : "v"(src), "i"(j * 512 + f * 64) : "memory");
    asm volatile("s_waitcnt vmcnt(32)" : "+v"(cv) :: "memory");
    const unsigned call = __builtin_amdgcn_readfirstlane(cv) + 1;
    if (lane == 0) asm volatile("global_store_dword %0, %1, off sc0 sc1\n\ts_nop 1" :: "v"(ctrp), "v"(call) : "memory");
    const unsigned par = call & 1;
    char* pws = P.ws[peer];
    char* dst = pws + par * PAR_BYTES + (size_t)P.rank * SRC_BYTES + (size_t)t * (NCH * PKT) + h * PKT + lane * 16;
    char* fl = pws + FLAG_OFF + par * FLAG_PAR + (size_t)P.rank * (16 * NCH * 16) + t * (NCH * 16) + h * 16;
    u32x4 fv = {call, 0x80u, call, 0u};
    PSTAMP(1);
#pragma unroll
    for (int j = 0; j < 8; ++j) {
      asm volatile("s_waitcnt vmcnt(%c4)" : "+v"(d[j][0]), "+v"(d[j][1]), "+v"(d[j][2]), "+v"(d[j][3]) : "i"(29 + (j > LAG ? j - LAG : 0)) : "memory");
      char* dj = dst + (size_t)j * (2 * PKT);
#pragma unroll
      for (int f = 0; f < 4; ++f)
        asm volatile("global_store_dwordx4 %0, %1, off offset:%c2 sc0 sc1\n\ts_nop 1" :: "v"(dj), "v"(d[j][f]), "i"(f * 1024) : "memory");
      if (j >= LAG) {
        asm volatile("s_waitcnt vmcnt(%c0)" :: "i"(after_cnt(j - LAG)) : "memory");
        if (lane == 0) asm volatile("global_store_dwordx4 %0, %1, off sc0 sc1\n\ts_nop 1" :: "v"(fl + (j - LAG) * 32), "v"(fv) : "memory");
        PSTAMP(2 + j - LAG);
      }
    }
#pragma unroll
    for (int j = 8 - LAG; j < 8; ++j) {
      asm volatile("s_waitcnt vmcnt(%c0)" :: "i"(after_cnt(j)) : "memory");
      if (lane == 0) asm volatile("global_store_dwordx4 %0, %1, off sc0 sc1\n\ts_nop 1" :: "v"(fl + j * 32), "v"(fv) : "memory");
      PSTAMP(2 + j);
    }
    return;
  }

  LDS_AS unsigned* cnt = (LDS_AS unsigned*)(lds + LDS_W);
  if (tid == 0) *cnt = 0;
  __builtin_amdgcn_s_waitcnt(0xC07F);
  __builtin_amdgcn_s_barrier();
  unsigned cv = __hip_atomic_load(ctrp, __ATOMIC_RELAXED, __HIP_MEMORY_SCOPE_SYSTEM);
  const int src_rank = xcd, rt = bi >> 3, nt = bi & 7;
  const int t = rt * 4 + wave;
  {
    const char* wsrc = P.w + (size_t)(wave * 512 + (lane >> 2)) * 512 + nt * 64 + (lane & 3) * 16;
#pragma unroll 8
    for (int i = 0; i < 32; ++i)
      __builtin_amdgcn_global_load_lds((const GLB_AS void*)(wsrc + (size_t)i * 16 * 512), (LDS_AS void*)(lds + (wave * 32 + i) * 1024), 16, 0, 0);
  }
  const unsigned call = __builtin_amdgcn_readfirstlane(cv) + 1;
  if (lane == 0) __hip_atomic_store(ctrp, call, __ATOMIC_RELAXED, __HIP_MEMORY_SCOPE_SYSTEM);
  __builtin_amdgcn_s_waitcnt(0x0F70);
  asm volatile("s_waitcnt vmcnt(0)" ::: "memory");
  if (lane == 0) __hip_atomic_fetch_add(cnt, 1u, __ATOMIC_RELEASE, __HIP_MEMORY_SCOPE_WORKGROUP);
  while (__hip_atomic_load(cnt, __ATOMIC_ACQUIRE, __HIP_MEMORY_SCOPE_WORKGROUP) < 4) __builtin_amdgcn_s_sleep(1);
  STAMP(1);

  const unsigned par = call & 1;
  const int g = lane >> 4, qq = (lane >> 2) & 3, pp = lane & 3;
  const int lbase = (8 * g + qq) * 64 + 16 * pp;
  f32x4 acc0 = {0.f, 0.f, 0.f, 0.f}, acc1 = {0.f, 0.f, 0.f, 0.f};
  bool bad = false;

  auto wfrag = [&](int c, int f, int tt) -> bf16x8 {
    const char* p = lds + c * 8192 + lbase + f * 2048 + tt * 8;
    s16x4 lo = __builtin_bit_cast(s16x4, __builtin_amdgcn_ds_read_tr16_b64_v4i16((LDS_AS s16x4*)(p)));
    s16x4 hi = __builtin_bit_cast(s16x4, __builtin_amdgcn_ds_read_tr16_b64_v4i16((LDS_AS s16x4*)(p + 256)));
    s16x8 r = __builtin_shufflevector(lo, hi, 0, 1, 2, 3, 4, 5, 6, 7);
    return __builtin_bit_cast(bf16x8, r);
  };

  if (src_rank == P.rank) {
    const char* xs = P.x + (size_t)(t * 16 + (lane & 15)) * 4096 + (lane >> 4) * 16;
#pragma unroll 2
    for (int c = 0; c < NCH; ++c) {
      u32x4 xv[4];
#pragma unroll
      for (int f = 0; f < 4; ++f) xv[f] = *(const u32x4*)(xs + c * 256 + f * 64);
#pragma unroll
      for (int f = 0; f < 4; ++f) {
        bf16x8 xb = __builtin_bit_cast(bf16x8, xv[f]);
        acc0 = __builtin_amdgcn_mfma_f32_16x16x32_bf16(wfrag(c, f, 0), xb, acc0, 0, 0, 0);
        acc1 = __builtin_amdgcn_mfma_f32_16x16x32_bf16(wfrag(c, f, 1), xb, acc1, 0, 0, 0);
      }
    }
  } else {
    char* slot = myws + par * PAR_BYTES + (size_t)src_rank * SRC_BYTES + (size_t)t * (NCH * PKT);
    const __amdgpu_buffer_rsrc_t drs = __builtin_amdgcn_make_buffer_rsrc(slot, 0, NCH * PKT, 0x00020000);
    char* flp = myws + FLAG_OFF + par * FLAG_PAR + (size_t)src_rank * (16 * NCH * 16) + t * (NCH * 16);
    const __amdgpu_buffer_rsrc_t frs = __builtin_amdgcn_make_buffer_rsrc(flp, 0, NCH * 16, 0x00020000);
    const int foff = (lane & 15) * 16;
    u32x4 fv = __builtin_amdgcn_raw_buffer_load_b128(frs, foff, 0, 17);
    unsigned long long t0 = __builtin_amdgcn_s_memrealtime();
#pragma unroll 1
    for (int c = 0; c < NCH; ++c) {
      bf16x8 wf[4][2];
#pragma unroll
      for (int f = 0; f < 4; ++f) { wf[f][0] = wfrag(c, f, 0); wf[f][1] = wfrag(c, f, 1); }
      while (true) {
        unsigned long long m = __ballot(fv.x == call && fv.z == call);
        if ((m >> c) & 1) break;
        if (__hip_atomic_load(tmo, __ATOMIC_RELAXED, __HIP_MEMORY_SCOPE_AGENT) == call ||
            __builtin_amdgcn_s_memrealtime() - t0 > DEADLINE) {
          __hip_atomic_store(tmo, call, __ATOMIC_RELAXED, __HIP_MEMORY_SCOPE_AGENT);
          bad = true; break;
        }
        __builtin_amdgcn_s_sleep(2);
        fv = __builtin_amdgcn_raw_buffer_load_b128(frs, foff, 0, 17);
      }
      if (bad) break;
      STAMP(2 + c);
      u32x4 xv[4];
#pragma unroll
      for (int f = 0; f < 4; ++f) xv[f] = __builtin_amdgcn_raw_buffer_load_b128(drs, lane * 16 + f * 1024, c * (int)PKT, 17);
      fv = __builtin_amdgcn_raw_buffer_load_b128(frs, foff, 0, 17);
#pragma unroll
      for (int f = 0; f < 4; ++f) {
        bf16x8 xb = __builtin_bit_cast(bf16x8, xv[f]);
        acc0 = __builtin_amdgcn_mfma_f32_16x16x32_bf16(wf[f][0], xb, acc0, 0, 0, 0);
        acc1 = __builtin_amdgcn_mfma_f32_16x16x32_bf16(wf[f][1], xb, acc1, 0, 0, 0);
      }
    }
  }
  STAMP(20);
  if (__hip_atomic_load(tmo, __ATOMIC_RELAXED, __HIP_MEMORY_SCOPE_AGENT) == call) bad = true;
  float o[8];
#pragma unroll
  for (int j = 0; j < 4; ++j) { o[j] = gelu(acc0[j]); o[4 + j] = gelu(acc1[j]); }
  u32x4 ov;
#pragma unroll
  for (int j = 0; j < 4; ++j) {
    f32x2 pr = {o[2 * j], o[2 * j + 1]};
    bf16x2 pb = __builtin_convertvector(pr, bf16x2);
    ov[j] = bad ? 0x7FC07FC0u : __builtin_bit_cast(unsigned, pb);
  }
  const int m = src_rank * 256 + t * 16 + (lane & 15);
  *(u32x4*)(P.out + (size_t)m * (NS * 2) + nt * 64 + g * 16) = ov;
  STAMP(21);
}
}

extern "C" void kernel_launch(int rank, int nranks, ncclComm_t comm, hipStream_t stream,
                              void* const* d_in, const int* in_sizes, int n_in,
                              void* d_out, int out_size, const int* in_dtypes, int out_dtype,
                              void* const* d_out_all, void* const* d_ws_all, size_t ws_bytes) {
  (void)comm; (void)in_sizes; (void)n_in; (void)out_size; (void)in_dtypes; (void)out_dtype; (void)d_out_all;
  if (nranks != NR || ws_bytes < WS_NEED || rank < 0 || rank >= NR) { fprintf(stderr, "bad config\n"); return; }
  static bool init = false;
  if (!init) {
    hipError_t e = hipFuncSetAttribute((const void*)ag_gemm, hipFuncAttributeMaxDynamicSharedMemorySize, LDS_BYTES);
    if (e != hipSuccess) { fprintf(stderr, "hipFuncSetAttribute: %s\n", hipGetErrorString(e)); return; }
    init = true;
  }
  Params P;
  P.x = (const char*)d_in[0]; P.w = (const char*)d_in[1]; P.out = (char*)d_out;
  for (int p = 0; p < NR; ++p) P.ws[p] = (char*)d_ws_all[p];
  P.rank = rank; P.pad = 0;
  hipLaunchKernelGGL(ag_gemm, dim3(256), dim3(320), LDS_BYTES, stream, P);
}
```

```cpp
#include <hip/hip_runtime.h>
#include <rccl/rccl.h>
#include <cstdio>
#include <cstdint>
#include <cstdlib>

typedef unsigned u32x4 __attribute__((ext_vector_type(4)));
typedef unsigned u32x2 __attribute__((ext_vector_type(2)));
typedef unsigned short u16x2 __attribute__((ext_vector_type(2)));
typedef short s16x4 __attribute__((ext_vector_type(4)));
typedef short s16x8 __attribute__((ext_vector_type(8)));
typedef __bf16 bf16x8 __attribute__((ext_vector_type(8)));
typedef __bf16 bf16x2 __attribute__((ext_vector_type(2)));
typedef float f32x4 __attribute__((ext_vector_type(4)));
typedef float f32x2 __attribute__((ext_vector_type(2)));
#define LDS_AS __attribute__((address_space(3)))
#define GLB_AS __attribute__((address_space(1)))

#define PMAP 0
#define POLL_SLEEP 1
#define NOCONS 0
#define OUTST 0
#define ROT 1
#define LASTWIN 7
#define FORCE_RAW 0

namespace {
constexpr int NR = 8;
constexpr int NS = 256;
constexpr int NCH = 16;
constexpr unsigned PKT = 4096;
constexpr size_t SRC_BYTES = 16 * NCH * PKT;
constexpr size_t PAR_BYTES = NR * SRC_BYTES;
constexpr size_t FLAG_OFF = 2 * PAR_BYTES;
constexpr size_t FLAG_SRC = 16 * NCH * 16;
constexpr size_t FLAG_PAR = NR * FLAG_SRC;
constexpr size_t CTR_OFF = FLAG_OFF + 2 * FLAG_PAR;
constexpr size_t TMO_OFF = CTR_OFF + 256 * 64;
constexpr size_t STAMP_OFF = FLAG_OFF + 256 * 1024;
constexpr size_t WS_NEED = FLAG_OFF + 512 * 1024;
constexpr unsigned long long DEADLINE = 20000000ull;
constexpr int LDS_W = 2048 * 64;
constexpr int LDS_BYTES = LDS_W + 64 + 8 * 256;

struct Params {
  const char* x; char* myws; const char* w; char* out;
  int rank; int pad;
  char* peer[14];
};

#define STAMP(i)
#define STAMP_DUMP()

__device__ __forceinline__ float gelu(float y) {
  const float c2 = 2.0f * 0.7978845608028654f * 1.4426950408889634f;
  float u = y * (c2 + (c2 * 0.044715f) * y * y);
  float e = __builtin_amdgcn_exp2f(-u);
  return y * __builtin_amdgcn_rcpf(1.0f + e);
}
__device__ __forceinline__ unsigned perm(unsigned s0, unsigned s1, unsigned sel) { return __builtin_amdgcn_perm(s0, s1, sel); }
__device__ __forceinline__ unsigned pkmax(unsigned a, unsigned b) {
  return __builtin_bit_cast(unsigned, __builtin_elementwise_max(__builtin_bit_cast(u16x2, a), __builtin_bit_cast(u16x2, b)));
}
__device__ __forceinline__ unsigned wave_umax(unsigned v) {
  v = max(v, (unsigned)__builtin_amdgcn_update_dpp(0, (int)v, 0x111, 0xf, 0xf, false));
  v = max(v, (unsigned)__builtin_amdgcn_update_dpp(0, (int)v, 0x112, 0xf, 0xf, false));
  v = max(v, (unsigned)__builtin_amdgcn_update_dpp(0, (int)v, 0x114, 0xf, 0xf, false));
  v = max(v, (unsigned)__builtin_amdgcn_update_dpp(0, (int)v, 0x118, 0xf, 0xf, false));
  v = max(v, (unsigned)__builtin_amdgcn_update_dpp(0, (int)v, 0x142, 0xa, 0xf, false));
  v = max(v, (unsigned)__builtin_amdgcn_update_dpp(0, (int)v, 0x143, 0xc, 0xf, false));
  return (unsigned)__builtin_amdgcn_readlane((int)v, 63);
}

__device__ __forceinline__ void decode(const u32x4& w0, const u32x4& w1, const u32x4& w2, unsigned ma, unsigned mb, int lane, u32x4 (&x)[4]) {
  const unsigned B = ma & 0x7Fu, B4 = B * 0x01010101u;
  unsigned H[8];
#pragma unroll
  for (int i = 0; i < 4; ++i) {
    const unsigned n = w2[i];
    H[2 * i] = B4 + (n & 0x87878787u);
    H[2 * i + 1] = B4 + (__builtin_amdgcn_alignbit(n, n, 4) & 0x87878787u);
  }
  const unsigned nexc = (ma >> 8) & 3u;
  if (nexc) {
    for (unsigned e = 0; e < nexc; ++e) {
      const unsigned rec = e == 0 ? (ma >> 16) : (e == 1 ? (mb & 0xFFFFu) : (mb >> 16));
      const unsigned el = rec & 63u, slot = (rec >> 6) & 31u, code = (rec >> 11) & 31u;
      const unsigned delta = code ? code : B;
      const unsigned v = ((unsigned)lane == el) ? (delta << (8 * (slot & 3u))) : 0u;
#pragma unroll
      for (int q = 0; q < 8; ++q) H[q] -= ((slot >> 2) == (unsigned)q) ? v : 0u;
    }
  }
#pragma unroll
  for (int p = 0; p < 8; ++p) {
    const unsigned L = p < 4 ? w0[p & 3] : w1[p & 3];
    x[p >> 1][(p & 1) * 2] = perm(H[p], L, 0x05010400u);
    x[p >> 1][(p & 1) * 2 + 1] = perm(H[p], L, 0x07030602u);
  }
}

#define WIN 18
#define FLAG(addr, val) do { if (lane < 7) asm volatile("global_store_dwordx4 %0, %1, off sc0 sc1\n\ts_nop 1" :: "v"(addr), "v"(val) : "memory"); } while (0)
#define PSTAMP(i)
#define STORE16(voff, data, sbase, imm) \
  asm volatile("global_store_dwordx4 %0, %1, %2 offset:%c3 sc0 sc1\n\ts_nop 1" :: "v"(voff), "v"(data), "s"(sbase), "i"(imm) : "memory")

__global__ void __launch_bounds__(320, 2) ag_gemm(Params P) {
  extern __shared__ __attribute__((aligned(16))) char lds[];
  asm volatile("" :: "s"(P.x), "s"(P.myws));
  const int tid = threadIdx.x;
  const int lane = tid & 63;
  const int wave = __builtin_amdgcn_readfirstlane(tid >> 6);
  const int b = blockIdx.x, xcd = b & 7, bi = b >> 3;
  const bool is_push = bi < 4;
  const int pidx = xcd * 4 + bi;
  const int ci = xcd * 28 + (bi - 4);
  unsigned* ctrp = (unsigned*)(P.myws + CTR_OFF + b * 64 + wave * 4);

  if (__builtin_expect(is_push, 1)) {
    if (wave != 0) return;
    const int t = pidx >> 1, h = pidx & 1;
    unsigned cv;
    asm volatile("global_load_dword %0, %1, off sc0 sc1" : "=v"(cv) : "v"(ctrp) : "memory");
    const char* src = P.x + (size_t)(t * 16 + (lane & 15)) * 4096 + h * 256 + (lane >> 4) * 16;
    const unsigned lb = (unsigned)(size_t)(LDS_AS char*)lds;
#define GLDS4(srcp, ldsb) do { _Pragma("unroll") for (int f_ = 0; f_ < 4; ++f_) { unsigned keep_; \
      asm volatile("s_mov_b32 %0, m0\n\ts_add_u32 m0, %2, %c3\n\ts_nop 0\n\tglobal_load_lds_dwordx4 %1, off offset:%c4\n\ts_mov_b32 m0, %0" \
                   : "=&s"(keep_) : "v"(srcp), "s"(ldsb), "i"(960 * f_), "i"(64 * f_) : "memory", "scc"); } } while (0)
    GLDS4(src, lb);
    GLDS4(src + 512, lb + 4096);
    const size_t doff = (size_t)P.rank * SRC_BYTES + (size_t)t * (NCH * PKT) + h * PKT;
    char* sb[7];
    { const int rot = pidx % 7;
#pragma unroll
      for (int q = 0; q < 7; ++q) sb[q] = P.peer[rot + q];
      asm volatile("" : "+s"(sb[0]), "+s"(sb[1]), "+s"(sb[2]), "+s"(sb[3]), "+s"(sb[4]), "+s"(sb[5]), "+s"(sb[6])); }
    char* fp = sb[0];
#pragma unroll
    for (int q = 1; q < 7; ++q) fp = lane == q ? sb[q] : fp;
    fp += FLAG_OFF + (size_t)P.rank * FLAG_SRC + t * (NCH * 16) + h * 16;
#pragma unroll
    for (int q = 0; q < 7; ++q) sb[q] += doff;
    asm volatile("" : "+s"(sb[0]), "+s"(sb[1]), "+s"(sb[2]), "+s"(sb[3]), "+s"(sb[4]), "+s"(sb[5]), "+s"(sb[6]), "+v"(fp));
    asm volatile("s_waitcnt vmcnt(4)" : "+v"(cv) :: "memory");
    const unsigned call = __builtin_amdgcn_readfirstlane(cv) + 1;
    {
      const unsigned cn = call;
      if (lane == 0) asm volatile("global_store_dword %0, %1, off sc0 sc1\n\ts_nop 1" :: "v"(ctrp), "v"(cn) : "memory");
    }
    const unsigned par = call & 1;
    fp += par * FLAG_PAR;
    const unsigned vbase = lane * 16 + par * (unsigned)PAR_BYTES;
    u32x4 fprev = {0u, 0u, 0u, 0u};
#pragma unroll 1
    for (int j = 0; j < 8; ++j) {
      if (j >= 1 && j < 7) GLDS4(src + (j + 1) * 512, lb + (j + 1) * 4096);
      if (j == 1) asm volatile("s_waitcnt vmcnt(26)" ::: "memory");
      unsigned D[16];
#pragma unroll
      for (int f = 0; f < 4; ++f) {
        u32x4 v = *(const LDS_AS u32x4*)((LDS_AS char*)lds + j * 4096 + f * 1024 + lane * 16);
        D[4 * f] = v[0]; D[4 * f + 1] = v[1]; D[4 * f + 2] = v[2]; D[4 * f + 3] = v[3];
      }
      unsigned L[8], H[8], T[8];
#pragma unroll
      for (int p = 0; p < 8; ++p) { L[p] = perm(D[2 * p + 1], D[2 * p], 0x06040200u); H[p] = perm(D[2 * p + 1], D[2 * p], 0x07050301u); T[p] = H[p] & 0x7F7F7F7Fu; }
      unsigned ma_ = T[0], mb_ = T[0] << 8;
#pragma unroll
      for (int p = 1; p < 8; ++p) { ma_ = pkmax(ma_, T[p]); mb_ = pkmax(mb_, T[p] << 8); }
      ma_ = pkmax(ma_, mb_);
      const unsigned Bmax = wave_umax(max(ma_ >> 24, (ma_ >> 8) & 0xFFu));
      const unsigned B = Bmax > 7u ? Bmax - 7u : 0u;
      const unsigned B4 = B * 0x01010101u;
      unsigned acc = 0xFFFFFFFFu;
#pragma unroll
      for (int p = 0; p < 8; ++p) { T[p] = (T[p] | 0x80808080u) - B4; acc &= T[p]; }
      unsigned metaA = B, metaB = 0;
      bool raw = FORCE_RAW;
      unsigned long long lm = __ballot((acc & 0x80808080u) != 0x80808080u);
      if (lm) {
        unsigned n = 0;
        do {
          const int el = __builtin_ctzll(lm); lm &= lm - 1;
#pragma unroll
          for (int q = 0; q < 8; ++q) {
            unsigned tq = (unsigned)__builtin_amdgcn_readlane((int)T[q], el);
            unsigned f = ~tq & 0x80808080u;
            if (f) {
              do {
                const unsigned sh = (unsigned)__builtin_ctz(f) - 7u; f &= f - 1;
                const unsigned delta = 128u - ((tq >> sh) & 0xFFu);
                unsigned code = delta;
                if (delta == B) code = 0;
                else if (delta > 31u) raw = true;
                const unsigned rec = (unsigned)el | ((4u * q + (sh >> 3)) << 6) | ((code & 31u) << 11);
                if (n == 0) metaA |= rec << 16; else if (n == 1) metaB |= rec; else if (n == 2) metaB |= rec << 16; else raw = true;
                ++n;
                tq = (tq & ~(0xFFu << sh)) | (0x80u << sh);
              } while (f);
              T[q] = lane == el ? tq : T[q];
            }
          }
        } while (lm && !raw);
        metaA |= (n & 3u) << 8;
      }
      const unsigned voff = vbase + j * (2 * PKT);
      char* fa = fp + (j - 1) * 32;
      if (!raw) {
        u32x4 w0 = {L[0], L[1], L[2], L[3]}, w1 = {L[4], L[5], L[6], L[7]}, w2;
#pragma unroll
        for (int i = 0; i < 4; ++i) {
          const unsigned ca = (H[2 * i] & 0x80808080u) | (T[2 * i] & 0x7F7F7F7Fu);
          const unsigned cb = (H[2 * i + 1] & 0x80808080u) | (T[2 * i + 1] & 0x7F7F7F7Fu);
          w2[i] = ca | __builtin_amdgcn_alignbit(cb, cb, 28);
        }
        if (j >= LASTWIN) {
#pragma unroll
          for (int q = 0; q < 7; ++q) { STORE16(voff, w0, sb[q], 0); STORE16(voff, w1, sb[q], 1024); STORE16(voff, w2, sb[q], 2048); }
          asm volatile("s_waitcnt vmcnt(21)" ::: "memory"); FLAG(fa, fprev); PSTAMP(2 + j - 1);
        } else
        {
#pragma unroll
        for (int q = 0; q < 7; ++q) {
          if (j >= 1) {
            asm volatile("s_waitcnt vmcnt(%c0)" :: "i"(WIN) : "memory");
            if (q == 6) { asm volatile("s_waitcnt vmcnt(18)" ::: "memory"); FLAG(fa, fprev); PSTAMP(2 + j - 1); }
          }
          STORE16(voff, w0, sb[q], 0); STORE16(voff, w1, sb[q], 1024); STORE16(voff, w2, sb[q], 2048);
        }
        }
      } else {
        metaA = 0x80u; metaB = 0;
        u32x4 r0 = {D[0], D[1], D[2], D[3]}, r1 = {D[4], D[5], D[6], D[7]}, r2 = {D[8], D[9], D[10], D[11]}, r3 = {D[12], D[13], D[14], D[15]};
#pragma unroll
        for (int q = 0; q < 7; ++q) { STORE16(voff, r0, sb[q], 0); STORE16(voff, r1, sb[q], 1024); STORE16(voff, r2, sb[q], 2048); STORE16(voff, r3, sb[q], 3072); }
        if (j >= 1) { asm volatile("s_waitcnt vmcnt(21)" ::: "memory"); FLAG(fa, fprev); PSTAMP(2 + j - 1); }
      }
      fprev = u32x4{call, metaA, call, metaB};
    }
    {
      asm volatile("s_waitcnt vmcnt(1)" ::: "memory");
      char* fa = fp + 7 * 32;
      FLAG(fa, fprev);
    }
    return;
  }

  LDS_AS unsigned* cnt = (LDS_AS unsigned*)(lds + LDS_W);
  if (tid == 0) *cnt = 0;
  __builtin_amdgcn_s_waitcnt(0xC07F);
  __builtin_amdgcn_s_barrier();
  const int grp = ci >> 3, nt = ci & 7;
  if (wave == 4 && grp >= 16) return;
  unsigned cv = __hip_atomic_load(ctrp, __ATOMIC_RELAXED, __HIP_MEMORY_SCOPE_SYSTEM);
  if (wave < 4) {
    const char* wsrc = P.w + (size_t)(wave * 512 + (lane >> 2)) * 512 + nt * 64 + (lane & 3) * 16;
#pragma unroll 8
    for (int i = 0; i < 32; ++i)
      __builtin_amdgcn_global_load_lds((const GLB_AS void*)(wsrc + (size_t)i * 16 * 512), (LDS_AS void*)(lds + (wave * 32 + i) * 1024), 16, 0, 0);
  }
  const unsigned call = __builtin_amdgcn_readfirstlane(cv) + 1;
  if (lane == 0) __hip_atomic_store(ctrp, call, __ATOMIC_RELAXED, __HIP_MEMORY_SCOPE_SYSTEM);
  __builtin_amdgcn_s_waitcnt(0x0F70);
  asm volatile("s_waitcnt vmcnt(0)" ::: "memory");
  if (wave < 4 && lane == 0) __hip_atomic_fetch_add(cnt, 1u, __ATOMIC_RELEASE, __HIP_MEMORY_SCOPE_WORKGROUP);
  while (__hip_atomic_load(cnt, __ATOMIC_ACQUIRE, __HIP_MEMORY_SCOPE_WORKGROUP) < 4) __builtin_amdgcn_s_sleep(1);
  STAMP(1);

  const unsigned par = call & 1;
  const int g = lane >> 4, qq = (lane >> 2) & 3, pp = lane & 3;
  const int lbase = (8 * g + qq) * 64 + 16 * pp;
  f32x4 acc0 = {0.f, 0.f, 0.f, 0.f}, acc1 = {0.f, 0.f, 0.f, 0.f};
  bool bad = false;
  unsigned* tmo = (unsigned*)(P.myws + TMO_OFF);

  auto wfrag = [&](int c, int f, int tt) -> u32x4 {
    const char* p = lds + c * 8192 + lbase + f * 2048 + tt * 8;
    u32x2 lo = __builtin_bit_cast(u32x2, __builtin_amdgcn_ds_read_tr16_b64_v4i16((LDS_AS s16x4*)(p)));
    u32x2 hi = __builtin_bit_cast(u32x2, __builtin_amdgcn_ds_read_tr16_b64_v4i16((LDS_AS s16x4*)(p + 256)));
    u32x4 r = {lo[0], lo[1], hi[0], hi[1]};
    return r;
  };
  auto mma = [&](const u32x4 (&wf)[4][2], const u32x4 (&xv)[4]) {
#pragma unroll
    for (int f = 0; f < 4; ++f) {
      bf16x8 xb = __builtin_bit_cast(bf16x8, xv[f]);
      acc0 = __builtin_amdgcn_mfma_f32_16x16x32_bf16(__builtin_bit_cast(bf16x8, wf[f][0]), xb, acc0, 0, 0, 0);
      acc1 = __builtin_amdgcn_mfma_f32_16x16x32_bf16(__builtin_bit_cast(bf16x8, wf[f][1]), xb, acc1, 0, 0, 0);
    }
  };

  int src_rank, t;
  unsigned tv = 0;
  if (wave == 4) {
    src_rank = P.rank; t = grp;
    const char* xs = P.x + (size_t)(t * 16 + (lane & 15)) * 4096 + (lane >> 4) * 16;
#pragma unroll 2
    for (int c = 0; c < NCH; ++c) {
      u32x4 xv[4];
#pragma unroll
      for (int f = 0; f < 4; ++f) xv[f] = *(const u32x4*)(xs + c * 256 + f * 64);
      u32x4 wf[4][2];
#pragma unroll
      for (int f = 0; f < 4; ++f) { wf[f][0] = wfrag(c, f, 0); wf[f][1] = wfrag(c, f, 1); }
      mma(wf, xv);
    }
  } else {
    src_rank = (P.rank + 1 + (grp >> 2)) & 7; t = (grp & 3) * 4 + wave;
    char* slot = P.myws + par * PAR_BYTES + (size_t)src_rank * SRC_BYTES + (size_t)t * (NCH * PKT);
    const __amdgpu_buffer_rsrc_t drs = __builtin_amdgcn_make_buffer_rsrc(slot, 0, NCH * PKT, 0x00020000);
    char* flp = P.myws + FLAG_OFF + par * FLAG_PAR + (size_t)src_rank * FLAG_SRC + t * (NCH * 16);
    const __amdgpu_buffer_rsrc_t frs = __builtin_amdgcn_make_buffer_rsrc(flp, 0, NCH * 16, 0x00020000);
    const int foff = (lane & 1) * 16;
    u32x4 fv = __builtin_amdgcn_raw_buffer_load_b128(frs, foff, 0, 17);
    const unsigned long long t0 = __builtin_amdgcn_s_memrealtime();
#pragma unroll 1
    for (int i = (NOCONS == 2 ? 7 : 0); i < 8; ++i) {
      u32x4 wa[4][2], wb[4][2];
#pragma unroll
      for (int f = 0; f < 4; ++f) { wa[f][0] = wfrag(2 * i, f, 0); wa[f][1] = wfrag(2 * i, f, 1); wb[f][0] = wfrag(2 * i + 1, f, 0); wb[f][1] = wfrag(2 * i + 1, f, 1); }
#pragma unroll
      for (int f = 0; f < 4; ++f) asm volatile("" : "+v"(wa[f][0]), "+v"(wa[f][1]), "+v"(wb[f][0]), "+v"(wb[f][1]));
      for (unsigned spin = 1;; ++spin) {
        const unsigned long long m = __ballot(fv.x == call && fv.z == call);
        if ((m & 3) == 3) break;
        if ((spin & 63u) == 0) {
          if (__hip_atomic_load(tmo, __ATOMIC_RELAXED, __HIP_MEMORY_SCOPE_AGENT) == call ||
              __builtin_amdgcn_s_memrealtime() - t0 > DEADLINE) {
            __hip_atomic_store(tmo, call, __ATOMIC_RELAXED, __HIP_MEMORY_SCOPE_AGENT);
            bad = true; break;
          }
        }
        __builtin_amdgcn_s_sleep(POLL_SLEEP);
        fv = __builtin_amdgcn_raw_buffer_load_b128(frs, foff, i * 32, 17);
      }
      if (bad) break;
      STAMP(2 + i);
      const unsigned a0 = __builtin_amdgcn_readlane((int)fv.y, 0), b0 = __builtin_amdgcn_readlane((int)fv.w, 0);
      const unsigned a1 = __builtin_amdgcn_readlane((int)fv.y, 1), b1 = __builtin_amdgcn_readlane((int)fv.w, 1);
      const int so = i * 2 * (int)PKT;
      if (!((a0 | a1) & 0x80u)) {
        u32x4 p0[3], p1[3];
#pragma unroll
        for (int f = 0; f < 3; ++f) p0[f] = __builtin_amdgcn_raw_buffer_load_b128(drs, lane * 16 + f * 1024, so, 17);
#pragma unroll
        for (int f = 0; f < 3; ++f) p1[f] = __builtin_amdgcn_raw_buffer_load_b128(drs, lane * 16 + f * 1024, so + (int)PKT, 17);
        fv = __builtin_amdgcn_raw_buffer_load_b128(frs, foff, ((i + 1) & 7) * 32, 17);
        tv = __hip_atomic_load(tmo, __ATOMIC_RELAXED, __HIP_MEMORY_SCOPE_AGENT);
        u32x4 xv[4];
        decode(p0[0], p0[1], p0[2], a0, b0, lane, xv);
        mma(wa, xv);
        decode(p1[0], p1[1], p1[2], a1, b1, lane, xv);
        mma(wb, xv);
      } else {
#pragma unroll 1
        for (int s = 0; s < 2; ++s) {
          const unsigned a = s ? a1 : a0, bb = s ? b1 : b0;
          u32x4 r[4], xv[4];
#pragma unroll
          for (int f = 0; f < 4; ++f) r[f] = __builtin_amdgcn_raw_buffer_load_b128(drs, lane * 16 + f * 1024, so + s * (int)PKT, 17);
          if (a & 0x80u) {
#pragma unroll
            for (int f = 0; f < 4; ++f) xv[f] = r[f];
          } else decode(r[0], r[1], r[2], a, bb, lane, xv);
          if (s) mma(wb, xv); else mma(wa, xv);
        }
        fv = __builtin_amdgcn_raw_buffer_load_b128(frs, foff, ((i + 1) & 7) * 32, 17);
        tv = __hip_atomic_load(tmo, __ATOMIC_RELAXED, __HIP_MEMORY_SCOPE_AGENT);
      }
    }
  }
  STAMP(20);
  if (tv == call) bad = true;
  float o[8];
#pragma unroll
  for (int j = 0; j < 4; ++j) { o[j] = gelu(acc0[j]); o[4 + j] = gelu(acc1[j]); }
  u32x4 ov;
#pragma unroll
  for (int j = 0; j < 4; ++j) {
    f32x2 pr = {o[2 * j], o[2 * j + 1]};
    bf16x2 pb = __builtin_convertvector(pr, bf16x2);
    ov[j] = bad ? 0x7FC07FC0u : __builtin_bit_cast(unsigned, pb);
  }
  const int m = src_rank * 256 + t * 16 + (lane & 15);
  {
    char* op = P.out + (size_t)m * (NS * 2) + nt * 64 + g * 16;
    *(u32x4*)op = ov;
  }
  STAMP(21);
  STAMP_DUMP();
}
}

extern "C" void kernel_launch(int rank, int nranks, ncclComm_t comm, hipStream_t stream,
                              void* const* d_in, const int* in_sizes, int n_in,
                              void* d_out, int out_size, const int* in_dtypes, int out_dtype,
                              void* const* d_out_all, void* const* d_ws_all, size_t ws_bytes) {
  (void)comm; (void)in_sizes; (void)n_in; (void)out_size; (void)in_dtypes; (void)out_dtype; (void)d_out_all;
  if (nranks != NR || ws_bytes < WS_NEED || rank < 0 || rank >= NR) { fprintf(stderr, "bad config\n"); return; }
  static bool init = false;
  if (!init) {
    hipError_t e = hipFuncSetAttribute((const void*)ag_gemm, hipFuncAttributeMaxDynamicSharedMemorySize, LDS_BYTES);
    if (e != hipSuccess) { fprintf(stderr, "hipFuncSetAttribute: %s\n", hipGetErrorString(e)); return; }
    init = true;
  }
  Params P;
  P.x = (const char*)d_in[0]; P.w = (const char*)d_in[1]; P.out = (char*)d_out; P.myws = (char*)d_ws_all[rank];
  for (int q = 0; q < 14; ++q) P.peer[q] = (char*)d_ws_all[(rank + 1 + q % 7) % NR];
  P.rank = rank; P.pad = 0;
  hipLaunchKernelGGL(ag_gemm, dim3(256), dim3(320), LDS_BYTES, stream, P);
}
```

```cpp
#include <hip/hip_runtime.h>
#include <rccl/rccl.h>
#include <cstdio>
#include <cstdint>

typedef unsigned u32x4 __attribute__((ext_vector_type(4)));
typedef unsigned u32x2 __attribute__((ext_vector_type(2)));
typedef unsigned short u16x2 __attribute__((ext_vector_type(2)));
typedef short s16x4 __attribute__((ext_vector_type(4)));
typedef __bf16 bf16x8 __attribute__((ext_vector_type(8)));
typedef __bf16 bf16x2 __attribute__((ext_vector_type(2)));
typedef float f32x4 __attribute__((ext_vector_type(4)));
typedef float f32x2 __attribute__((ext_vector_type(2)));
#define LDS_AS __attribute__((address_space(3)))
#define GLB_AS __attribute__((address_space(1)))

#define POLL_SLEEP 1
#define FORCE_RAW 0
#define OUTST 0
#define WARM 0
#define WARM_SLEEP 0
#define DAUX 17
#define STAUX "sc0 sc1"
#define LASTWIN 1
#define WINA 12

namespace {
constexpr int NR = 8;
constexpr int NS = 256;
constexpr int NCH = 16;
constexpr unsigned PKT = 4096;
constexpr size_t SRC_BYTES = 16 * NCH * PKT;
constexpr size_t PAR_BYTES = NR * SRC_BYTES;
constexpr size_t FLAG_OFF = 2 * PAR_BYTES;
constexpr unsigned HDR = 128;
constexpr size_t FLAG_SRC = 16 * NCH * HDR;
constexpr size_t FLAG_PAR = NR * FLAG_SRC;
constexpr size_t CTR_OFF = FLAG_OFF + 2 * FLAG_PAR;
constexpr size_t TMO_OFF = CTR_OFF + 256 * 64;
constexpr size_t STAMP_OFF = FLAG_OFF + 640 * 1024;
constexpr size_t WS_NEED = FLAG_OFF + 896 * 1024;
constexpr unsigned long long DEADLINE = 20000000ull;
constexpr int LDS_W = 2048 * 64;
constexpr int LDS_SYNC = LDS_W;
constexpr int LDS_COR = LDS_W + 64;
constexpr int LDS_BYTES = LDS_COR + 16 * 1024;
constexpr int PUSH_LDS = 40 * 1024;
constexpr int LDS_HDR = 32 * 1024;
constexpr unsigned NREC0 = 22, NREC1 = 46;
constexpr int THREADS = 576;

struct Params {
  const char* x; char* myws; const char* w; char* out;
  int rank; int pad;
  char* peer[14];
};

__device__ __forceinline__ f32x2 gelu2(f32x2 y) {
  const float c2 = -2.0f * 0.7978845608028654f * 1.4426950408889634f;
  const f32x2 k1 = {c2, c2}, k3 = {c2 * 0.044715f, c2 * 0.044715f}, one = {1.0f, 1.0f};
  const f32x2 u = y * (k1 + k3 * (y * y));
  const f32x2 d = one + f32x2{__builtin_amdgcn_exp2f(u[0]), __builtin_amdgcn_exp2f(u[1])};
  return y * f32x2{__builtin_amdgcn_rcpf(d[0]), __builtin_amdgcn_rcpf(d[1])};
}
__device__ __forceinline__ unsigned perm(unsigned s0, unsigned s1, unsigned sel) { return __builtin_amdgcn_perm(s0, s1, sel); }
__device__ __forceinline__ unsigned pkmax(unsigned a, unsigned b) {
  return __builtin_bit_cast(unsigned, __builtin_elementwise_max(__builtin_bit_cast(u16x2, a), __builtin_bit_cast(u16x2, b)));
}
__device__ __forceinline__ unsigned wave_umax(unsigned v) {
  v = max(v, (unsigned)__builtin_amdgcn_update_dpp(0, (int)v, 0x111, 0xf, 0xf, false));
  v = max(v, (unsigned)__builtin_amdgcn_update_dpp(0, (int)v, 0x112, 0xf, 0xf, false));
  v = max(v, (unsigned)__builtin_amdgcn_update_dpp(0, (int)v, 0x114, 0xf, 0xf, false));
  v = max(v, (unsigned)__builtin_amdgcn_update_dpp(0, (int)v, 0x118, 0xf, 0xf, false));
  v = max(v, (unsigned)__builtin_amdgcn_update_dpp(0, (int)v, 0x142, 0xa, 0xf, false));
  v = max(v, (unsigned)__builtin_amdgcn_update_dpp(0, (int)v, 0x143, 0xc, 0xf, false));
  return (unsigned)__builtin_amdgcn_readlane((int)v, 63);
}

typedef unsigned u32x3 __attribute__((ext_vector_type(3)));
__device__ __forceinline__ void decode_hi(const u32x3& w2, unsigned B4, unsigned (&H)[8]) {
#pragma unroll
  for (int k = 0; k < 3; ++k) {
    const unsigned n = w2[k];
    H[2 * k] = B4 + (n & 0x83838383u);
    H[2 * k + 1] = B4 + (__builtin_amdgcn_alignbit(n, n, 4) & 0x83838383u);
  }
  H[6] = B4 + (((w2[0] >> 2) & 0x01010101u) | ((w2[1] >> 1) & 0x02020202u) | ((w2[2] << 5) & 0x80808080u));
  H[7] = B4 + (((w2[0] >> 6) & 0x01010101u) | ((w2[1] >> 5) & 0x02020202u) | ((w2[2] << 1) & 0x80808080u));
}
__device__ __forceinline__ u32x4 expand(const unsigned (&H)[8], const u32x4& w0, const u32x4& w1, int f, unsigned o80) {
  const unsigned La = f < 2 ? w0[(f & 1) * 2] : w1[(f & 1) * 2], Lb = f < 2 ? w0[(f & 1) * 2 + 1] : w1[(f & 1) * 2 + 1];
  return u32x4{perm(H[2 * f], La, 0x05010400u) - o80, perm(H[2 * f], La, 0x07030602u) - o80,
               perm(H[2 * f + 1], Lb, 0x05010400u) - o80, perm(H[2 * f + 1], Lb, 0x07030602u) - o80};
}

#define STORE16(voff, data, sbase, imm) \
  asm volatile("global_store_dwordx4 %0, %1, %2 offset:%c3 " STAUX "\n\ts_nop 1" :: "v"(voff), "v"(data), "s"(sbase), "i"(imm) : "memory")
#define STORE12(voff, data, sbase, imm) \
  asm volatile("global_store_dwordx3 %0, %1, %2 offset:%c3 " STAUX "\n\ts_nop 1" :: "v"(voff), "v"(data), "s"(sbase), "i"(imm) : "memory")
#define STAMP_PARAM
#define STAMP_ARG
#define WAITV(n) asm volatile("s_waitcnt vmcnt(%c0)" :: "i"(n) : "memory")

#define GLDS4(srcp, ldsb) do { _Pragma("unroll") for (int f_ = 0; f_ < 4; ++f_) { unsigned keep_; \
      asm volatile("s_mov_b32 %0, m0\n\ts_add_u32 m0, %2, %c3\n\ts_nop 0\n\tglobal_load_lds_dwordx4 %1, off offset:%c4\n\ts_mov_b32 m0, %0" \
                   : "=&s"(keep_) : "v"(srcp), "s"(ldsb), "i"(960 * f_), "i"(64 * f_) : "memory", "scc"); } } while (0)

struct Window { unsigned o, B; bool nofit; };
__device__ __forceinline__ Window pick_window(const unsigned (&D)[16]) {
  unsigned mx = D[0] & 0x7FFF7FFFu;
#pragma unroll
  for (int i = 1; i < 16; ++i) mx = pkmax(mx, D[i] & 0x7FFF7FFFu);
  const unsigned el = max(mx >> 16, mx & 0xFFFFu) >> 7;
  const unsigned emax = wave_umax(el);
  const unsigned ntop = (unsigned)__builtin_popcountll(__ballot(el == emax));
  const unsigned etop = (ntop <= 12u && emax > 0u) ? emax - 1u : emax;
  Window w;
  w.o = (etop & 1u) ^ 1u;
  const unsigned Et = (etop + w.o) >> 1;
  w.B = Et > 3u ? Et - 3u : 0u;
  w.nofit = FORCE_RAW || emax == 255u;
  return w;
}
__device__ __forceinline__ void read_packet(const LDS_AS char* lds, int j, int lane, unsigned (&D)[16]) {
#pragma unroll
  for (int f = 0; f < 4; ++f) {
    const u32x4 v = *(const LDS_AS u32x4*)(lds + j * 4096 + f * 1024 + lane * 16);
    D[4 * f] = v[0]; D[4 * f + 1] = v[1]; D[4 * f + 2] = v[2]; D[4 * f + 3] = v[3];
  }
}

template <bool HI>
__device__ __forceinline__ void push(const Params& P, LDS_AS char* lds, volatile LDS_AS unsigned* ack, unsigned* ctrp, int pidx, int lane STAMP_PARAM) {
    const int t = pidx >> 1, h = pidx & 1;
    unsigned cv;
    asm volatile("global_load_dword %0, %1, off sc0 sc1" : "=v"(cv) : "v"(ctrp) : "memory");
    const char* src = P.x + (size_t)(t * 16 + (lane & 15)) * 4096 + h * 256 + (lane >> 4) * 16;
    const unsigned lb = (unsigned)(size_t)lds;
    GLDS4(src, lb);
    GLDS4(src + 512, lb + 4096);
    if (!HI && lane == 0) *ack = 0;
    __builtin_amdgcn_s_waitcnt(0xC07F);
    __builtin_amdgcn_s_barrier();
    const size_t doff = (size_t)P.rank * SRC_BYTES + (size_t)t * (NCH * PKT) + h * PKT;
    char* sb[7];
    { const int rot = pidx % 7;
#pragma unroll
      for (int q = 0; q < 7; ++q) { sb[q] = P.peer[rot + q]; asm volatile("" : "+s"(sb[q])); } }
    char* fp = sb[0];
    if (HI) {
#pragma unroll
      for (int q = 1; q < 7; ++q) fp = (lane >> 3) == q ? sb[q] : fp;
      fp += FLAG_OFF + (size_t)P.rank * FLAG_SRC + t * (NCH * HDR) + h * HDR + (lane & 7) * 16;
      asm volatile("" : "+v"(fp));
    }
#pragma unroll
    for (int q = 0; q < 7; ++q) { sb[q] += doff; asm volatile("" : "+s"(sb[q])); }
    asm volatile("s_waitcnt vmcnt(4)" : "+v"(cv) :: "memory");
    const unsigned call = __builtin_amdgcn_readfirstlane(cv) + 1;
    {
      const unsigned cn = call;
      if (lane == 0) asm volatile("global_store_dword %0, %1, off sc0 sc1\n\ts_nop 1" :: "v"(ctrp), "v"(cn) : "memory");
    }
    const unsigned par = call & 1;
    const unsigned vbase = lane * 16 + par * (unsigned)PAR_BYTES;

    if (!HI) {
#pragma unroll 1
      for (int j = 0; j < 8; ++j) {
        if (j >= 1 && j < 7) GLDS4(src + (j + 1) * 512, lb + (j + 1) * 4096);
        if (j == 1) WAITV(19);
        unsigned D[16];
        read_packet(lds, j, lane, D);
        const Window wd = pick_window(D);
        if (wd.nofit) {
          WAITV(0);
          if (lane == 0) *ack = j + 1;
          if (j == 0) { WAITV(0); }
          continue;
        }
        const unsigned o80 = wd.o * 0x00800080u, voff = vbase + j * (2 * PKT);
        u32x4 w0, w1;
#pragma unroll
        for (int p = 0; p < 4; ++p) {
          w0[p] = perm(D[2 * p + 1] + o80, D[2 * p] + o80, 0x06040200u);
          w1[p] = perm(D[2 * p + 9] + o80, D[2 * p + 8] + o80, 0x06040200u);
        }
        if (j >= 1 && (j < 7 || LASTWIN)) {
#pragma unroll
          for (int q = 0; q < 7; ++q) {
            WAITV(WINA);
            if (q == 6) { WAITV(12); if (lane == 0) *ack = j; }
            STORE16(voff, w0, sb[q], 0); STORE16(voff, w1, sb[q], 1024);
          }
        } else {
#pragma unroll
          for (int q = 0; q < 7; ++q) { STORE16(voff, w0, sb[q], 0); STORE16(voff, w1, sb[q], 1024); }
          if (j == 7) { WAITV(14); if (lane == 0) *ack = 7; }
        }
      }
      WAITV(0);
      if (lane == 0) *ack = 8;
      return;
    }

    fp += par * FLAG_PAR;
    LDS_AS char* hdr = lds + LDS_HDR;
    unsigned nprev = 0;
    auto wait_ack = [&](unsigned v) { while (*ack < v) __builtin_amdgcn_s_sleep(1); };
    auto send_header = [&](int jj, unsigned nrec) {
      const LDS_AS unsigned* img = (const LDS_AS unsigned*)(hdr + (jj & 1) * 128) + 3 * (lane & 7);
      const u32x4 v = {call, img[0], img[1], img[2]};
      char* fa = fp + jj * (2 * HDR);
      if (lane < 56 && (lane & 7) < (nrec > NREC0 ? 8 : 4))
        asm volatile("global_store_dwordx4 %0, %1, off sc0 sc1\n\ts_nop 1" :: "v"(fa), "v"(v) : "memory");
    };
#pragma unroll 1
    for (int j = 0; j < 8; ++j) {
      if (j >= 1 && j < 7) GLDS4(src + (j + 1) * 512, lb + (j + 1) * 4096);
      if (j == 1) WAITV(12);
      unsigned D[16];
      read_packet(lds, j, lane, D);
      const Window wd = pick_window(D);
      const unsigned B4 = wd.B * 0x01010101u, o80 = wd.o * 0x00800080u;
      bool raw = wd.nofit;
      const unsigned voff = vbase + j * (2 * PKT);
      unsigned n = 0;
      if (!raw) {
        unsigned U[8], c[8], em = 0;
#pragma unroll
        for (int p = 0; p < 8; ++p) {
          const unsigned H = perm(D[2 * p + 1] + o80, D[2 * p] + o80, 0x07050301u);
          const unsigned u = (H | 0x80808080u) - B4;
          const unsigned sm = (u & 0x7C7C7C7Cu) + 0x7C7C7C7Cu;
          em |= ((~u | sm) & 0x80808080u) >> (7 - p);
          c[p] = (H & 0x80808080u) | (u & 0x03030303u);
          U[p] = u;
        }
        LDS_AS char* lst = hdr + (j & 1) * 128;
        unsigned long long bm = __ballot(em != 0);
        if (bm) {
          unsigned ovm = 0;
          LDS_AS char* dummy = hdr + 256 + lane * 2;
          do {
            const bool has = em != 0;
            const unsigned sp = (unsigned)__builtin_ctz(em | 0x80000000u);
            const bool s1 = sp & 1u, s2 = sp & 2u, s4 = sp & 4u;
            const unsigned ua = s1 ? U[1] : U[0], ub = s1 ? U[3] : U[2], uc = s1 ? U[5] : U[4], ud = s1 ? U[7] : U[6];
            const unsigned ue = s2 ? ub : ua, uf = s2 ? ud : uc;
            const unsigned code = 40u - __builtin_amdgcn_ubfe(s4 ? uf : ue, (sp & 24u) | 2u, 6u);
            ovm = max(ovm, has ? code : 0u);
            const unsigned rec = (unsigned)lane | ((sp & 7u) << 8) | ((sp >> 3) << 6) | (code << 11);
            const unsigned pos = n + __builtin_amdgcn_mbcnt_hi((unsigned)(bm >> 32), __builtin_amdgcn_mbcnt_lo((unsigned)bm, 0u));
            LDS_AS char* dst = (has && pos < NREC1) ? lst + 4 + 2 * pos : dummy;
            *(LDS_AS unsigned short*)dst = (unsigned short)rec;
            n += (unsigned)__builtin_popcountll(bm);
            em &= em - 1;
            bm = __ballot(em != 0);
          } while (bm && n <= NREC1);
          if (__ballot(ovm > 31u)) raw = true;
        }
        if (n > NREC1) raw = true;
        if (!raw) {
          u32x3 w2;
          w2[0] = c[0] | __builtin_amdgcn_alignbit(c[1], c[1], 28) | ((c[6] & 0x01010101u) << 2) | ((c[7] & 0x01010101u) << 6);
          w2[1] = c[2] | __builtin_amdgcn_alignbit(c[3], c[3], 28) | ((c[6] & 0x02020202u) << 1) | ((c[7] & 0x02020202u) << 5);
          w2[2] = c[4] | __builtin_amdgcn_alignbit(c[5], c[5], 28) | ((c[6] & 0x80808080u) >> 5) | ((c[7] & 0x80808080u) >> 1);
          if (lane == 0) *(LDS_AS unsigned*)lst = wd.B | (wd.o << 7) | (n << 16);
          const unsigned voff12 = voff - lane * 4;
#pragma unroll
          for (int q = 0; q < 7; ++q) STORE12(voff12, w2, sb[q], 2048);
        }
      }
      if (raw) {
        wait_ack(j + 1);
        n = 0;
        if (lane == 0) *(LDS_AS unsigned*)(hdr + (j & 1) * 128) = 0x100u;
        const u32x4 r0 = {D[0], D[1], D[2], D[3]}, r1 = {D[4], D[5], D[6], D[7]}, r2 = {D[8], D[9], D[10], D[11]}, r3 = {D[12], D[13], D[14], D[15]};
#pragma unroll
        for (int q = 0; q < 7; ++q) { STORE16(voff, r0, sb[q], 0); STORE16(voff, r1, sb[q], 1024); STORE16(voff, r2, sb[q], 2048); STORE16(voff, r3, sb[q], 3072); }
      }
      if (j >= 1) {
        WAITV(7);
        wait_ack(j);
        send_header(j - 1, nprev);
      }
      nprev = n;
    }
    WAITV(1);
    wait_ack(8);
    send_header(7, nprev);
}

__global__ void __launch_bounds__(THREADS, 3) ag_gemm(Params P) {
  extern __shared__ __attribute__((aligned(16))) char lds[];
  asm volatile("" :: "s"(P.x), "s"(P.myws));
  const int tid = threadIdx.x;
  const int lane = tid & 63;
  const int wave = __builtin_amdgcn_readfirstlane(tid >> 6);
  const int b = blockIdx.x, xcd = b & 7, bi = b >> 3;
  const bool is_push = bi < 4;
  unsigned* ctrp = (unsigned*)(P.myws + CTR_OFF + b * 64 + wave * 4);

  if (__builtin_expect(is_push, 1)) {
    const int pidx = xcd * 4 + bi;
    volatile LDS_AS unsigned* ack = (volatile LDS_AS unsigned*)((LDS_AS char*)lds + 2 * PUSH_LDS);
    if (wave == 0) push<false>(P, (LDS_AS char*)lds, ack, ctrp, pidx, lane STAMP_ARG);
    else if (wave == 1) push<true>(P, (LDS_AS char*)lds + PUSH_LDS, ack, ctrp, pidx, lane STAMP_ARG);
    return;
  }

  const int ci = xcd * 28 + (bi - 4);
  LDS_AS unsigned* sync = (LDS_AS unsigned*)(lds + LDS_SYNC);
  if (tid < 16) sync[tid] = 0;
  __builtin_amdgcn_s_waitcnt(0xC07F);
  __builtin_amdgcn_s_barrier();
  const int grp = ci >> 3, nt = ci & 7;
  const bool warm_cu = WARM && (nt & 3) == 0;
  if (wave == 8 && grp >= 16 && !warm_cu) return;
  unsigned cv = __hip_atomic_load(ctrp, __ATOMIC_RELAXED, __HIP_MEMORY_SCOPE_SYSTEM);
  if (wave < 8) {
    const char* wsrc = P.w + (size_t)(wave * 256 + (lane >> 2)) * 512 + nt * 64 + (lane & 3) * 16;
#pragma unroll 8
    for (int i = 0; i < 16; ++i)
      __builtin_amdgcn_global_load_lds((const GLB_AS void*)(wsrc + (size_t)i * 16 * 512), (LDS_AS void*)(lds + (wave * 16 + i) * 1024), 16, 0, 0);
  }
  const unsigned call = __builtin_amdgcn_readfirstlane(cv) + 1;
  if (lane == 0) __hip_atomic_store(ctrp, call, __ATOMIC_RELAXED, __HIP_MEMORY_SCOPE_SYSTEM);
  __builtin_amdgcn_s_waitcnt(0x0F70);
  asm volatile("s_waitcnt vmcnt(0)" ::: "memory");
  if (wave < 8 && lane == 0) __hip_atomic_fetch_add(sync, 1u, __ATOMIC_RELEASE, __HIP_MEMORY_SCOPE_WORKGROUP);
  while (__hip_atomic_load(sync, __ATOMIC_ACQUIRE, __HIP_MEMORY_SCOPE_WORKGROUP) < 8) __builtin_amdgcn_s_sleep(1);

  const unsigned par = call & 1;
  const int g = lane >> 4, qq = (lane >> 2) & 3, pp = lane & 3;
  const int lbase = (8 * g + qq) * 64 + 16 * pp;
  f32x4 acc0 = {0.f, 0.f, 0.f, 0.f}, acc1 = {0.f, 0.f, 0.f, 0.f};

  auto wfrag = [&](int c, int f, int tt) -> u32x4 {
    const char* p = lds + c * 8192 + lbase + f * 2048 + tt * 8;
    u32x2 lo = __builtin_bit_cast(u32x2, __builtin_amdgcn_ds_read_tr16_b64_v4i16((LDS_AS s16x4*)(p)));
    u32x2 hi = __builtin_bit_cast(u32x2, __builtin_amdgcn_ds_read_tr16_b64_v4i16((LDS_AS s16x4*)(p + 256)));
    return u32x4{lo[0], lo[1], hi[0], hi[1]};
  };
  auto mma1 = [&](const u32x4& wa_, const u32x4& wb_, const u32x4& x) {
    const bf16x8 xb = __builtin_bit_cast(bf16x8, x);
    acc0 = __builtin_amdgcn_mfma_f32_16x16x32_bf16(__builtin_bit_cast(bf16x8, wa_), xb, acc0, 0, 0, 0);
    acc1 = __builtin_amdgcn_mfma_f32_16x16x32_bf16(__builtin_bit_cast(bf16x8, wb_), xb, acc1, 0, 0, 0);
  };
  auto cvt2 = [&](float a, float c) -> unsigned {
    const f32x2 pr = gelu2(f32x2{a, c});
    return __builtin_bit_cast(unsigned, __builtin_convertvector(pr, bf16x2));
  };

  if (wave == 8) {
    if (grp < 16) {
      const int t = grp;
      const char* xs = P.x + (size_t)(t * 16 + (lane & 15)) * 4096 + (lane >> 4) * 16;
#pragma unroll 2
      for (int c = 0; c < NCH; ++c) {
        u32x4 xv[4];
#pragma unroll
        for (int f = 0; f < 4; ++f) xv[f] = *(const u32x4*)(xs + c * 256 + f * 64);
#pragma unroll
        for (int f = 0; f < 4; ++f) mma1(wfrag(c, f, 0), wfrag(c, f, 1), xv[f]);
      }
      const u32x4 ov = {cvt2(acc0[0], acc0[1]), cvt2(acc0[2], acc0[3]), cvt2(acc1[0], acc1[1]), cvt2(acc1[2], acc1[3])};
      *(u32x4*)(P.out + (size_t)(P.rank * 256 + t * 16 + (lane & 15)) * (NS * 2) + nt * 64 + g * 16) = ov;
    }
    if (warm_cu) {
      const int ps = (P.rank + 1 + (grp >> 2)) & 7, t0_ = (grp & 3) * 4;
      char* flp = P.myws + FLAG_OFF + par * FLAG_PAR + (size_t)ps * FLAG_SRC + t0_ * (NCH * HDR);
      const __amdgpu_buffer_rsrc_t frs = __builtin_amdgcn_make_buffer_rsrc(flp, 0, 4 * NCH * 16, 0x00020000);
      const int l16 = lane & 15;
      const int foff = l16 < 8 ? (l16 >> 1) * 256 + (l16 & 1) * 128 : ((l16 - 8) >> 1) * 256 + (14 + (l16 & 1)) * 16;
      const unsigned long long tq0 = __builtin_amdgcn_s_memrealtime();
      for (unsigned spin = 1;; ++spin) {
        const unsigned v = __builtin_amdgcn_raw_buffer_load_b32(frs, foff, 0, 17);
        if ((__ballot(v == call) & 0xFF00) == 0xFF00) break;
        if ((spin & 255u) == 0 && __builtin_amdgcn_s_memrealtime() - tq0 > DEADLINE) break;
      }
    }
    return;
  }

  const int h = wave & 1;
  const int src_rank = (P.rank + 1 + (grp >> 2)) & 7, t = (grp & 3) * 4 + (wave >> 1);
  unsigned* tmo = (unsigned*)(P.myws + TMO_OFF);
  bool bad = false;
  unsigned tv = 0;
  {
    char* slot = P.myws + par * PAR_BYTES + (size_t)src_rank * SRC_BYTES + (size_t)t * (NCH * PKT) + h * PKT;
    const __amdgpu_buffer_rsrc_t drs = __builtin_amdgcn_make_buffer_rsrc(slot, 0, NCH * PKT - h * PKT, 0x00020000);
    char* flp = P.myws + FLAG_OFF + par * FLAG_PAR + (size_t)src_rank * FLAG_SRC + t * (NCH * HDR) + h * HDR;
    const __amdgpu_buffer_rsrc_t frs = __builtin_amdgcn_make_buffer_rsrc(flp, 0, NCH * HDR - h * HDR, 0x00020000);
    const int foff = (lane & 31) * 4;
    const unsigned rec0 = (lane < 32 && (lane & 3) != 0 && lane != 1) ? 2u * (3u * (lane >> 2) + (lane & 3) - 1u) - 2u : 255u;
    LDS_AS char* cor = (LDS_AS char*)lds + LDS_COR + wave * 2048;
    *(LDS_AS u32x4*)(cor + lane * 32) = u32x4{0u, 0u, 0u, 0u};
    *(LDS_AS u32x4*)(cor + lane * 32 + 16) = u32x4{0u, 0u, 0u, 0u};
    const unsigned long long t0 = __builtin_amdgcn_s_memrealtime();
    unsigned fv = __builtin_amdgcn_raw_buffer_load_b32(frs, foff, 0, 17);
#pragma unroll 1
    for (int j = 0; j < 8; ++j) {
      const int c = 2 * j + h;
      u32x4 wf[4][2];
#pragma unroll
      for (int k = 0; k < 4; ++k) { wf[k][0] = wfrag(c, k, 0); wf[k][1] = wfrag(c, k, 1); }
#pragma unroll
      for (int k = 0; k < 4; ++k) asm volatile("" : "+v"(wf[k][0]), "+v"(wf[k][1]));
      unsigned meta;
      for (unsigned spin = 1;; ++spin) {
        const unsigned long long m = __ballot(fv == call);
        meta = (unsigned)__builtin_amdgcn_readlane((int)fv, 1);
        if ((m & 0x1111u) == 0x1111u && (((meta >> 16) & 63u) <= NREC0 || (m & 0x11110000u) == 0x11110000u)) break;
        if ((spin & 1023u) == 0) {
          if (__hip_atomic_load(tmo, __ATOMIC_RELAXED, __HIP_MEMORY_SCOPE_AGENT) == call ||
              __builtin_amdgcn_s_memrealtime() - t0 > DEADLINE) {
            __hip_atomic_store(tmo, call, __ATOMIC_RELAXED, __HIP_MEMORY_SCOPE_AGENT);
            bad = true; break;
          }
        }
        __builtin_amdgcn_s_sleep(POLL_SLEEP);
        fv = __builtin_amdgcn_raw_buffer_load_b32(frs, foff, j * (int)(2 * HDR), 17);
      }
      if (bad) break;
      const int so = j * 2 * (int)PKT;
      if (__builtin_expect(meta & 0x100u, 0)) {
        const u32x4 r0 = __builtin_amdgcn_raw_buffer_load_b128(drs, lane * 16, so, DAUX);
        const u32x4 r1 = __builtin_amdgcn_raw_buffer_load_b128(drs, lane * 16 + 1024, so, DAUX);
        const u32x4 r2 = __builtin_amdgcn_raw_buffer_load_b128(drs, lane * 16 + 2048, so, DAUX);
        const u32x4 r3 = __builtin_amdgcn_raw_buffer_load_b128(drs, lane * 16 + 3072, so, DAUX);
        fv = __builtin_amdgcn_raw_buffer_load_b32(frs, foff, ((j + 1) & 7) * (int)(2 * HDR), 17);
        tv = __hip_atomic_load(tmo, __ATOMIC_RELAXED, __HIP_MEMORY_SCOPE_AGENT);
        mma1(wf[0][0], wf[0][1], r0); mma1(wf[1][0], wf[1][1], r1); mma1(wf[2][0], wf[2][1], r2); mma1(wf[3][0], wf[3][1], r3);
      } else {
        const unsigned hv = fv;
        asm volatile("" :: "s"(meta), "v"(hv) : "memory");
        const u32x3 w2 = __builtin_amdgcn_raw_buffer_load_b96(drs, lane * 12 + 2048, so, DAUX);
        const u32x4 w0 = __builtin_amdgcn_raw_buffer_load_b128(drs, lane * 16, so, DAUX);
        const u32x4 w1 = __builtin_amdgcn_raw_buffer_load_b128(drs, lane * 16 + 1024, so, DAUX);
        const unsigned fnext = __builtin_amdgcn_raw_buffer_load_b32(frs, foff, ((j + 1) & 7) * (int)(2 * HDR), 17);
        tv = __hip_atomic_load(tmo, __ATOMIC_RELAXED, __HIP_MEMORY_SCOPE_AGENT);
        const unsigned B = meta & 0x7Fu, nrec = (meta >> 16) & 63u;
#pragma unroll
        for (int e = 0; e < 2; ++e) {
          const unsigned rec = e ? hv >> 16 : hv & 0xFFFFu;
          const unsigned slot = (rec >> 6) & 31u;
          const unsigned d = ((rec >> 11) - 8u) * 4u;
          if (rec0 + e < nrec)
            __hip_atomic_fetch_add((LDS_AS unsigned*)(cor + (rec & 63u) * 32 + (slot >> 2) * 4), d << (8 * (slot & 3u)), __ATOMIC_RELAXED, __HIP_MEMORY_SCOPE_WORKGROUP);
        }
        asm volatile("" ::: "memory");
        const u32x4 c0 = *(LDS_AS u32x4*)(cor + lane * 32), c1 = *(LDS_AS u32x4*)(cor + lane * 32 + 16);
        asm volatile("" ::: "memory");
        *(LDS_AS u32x4*)(cor + lane * 32) = u32x4{0u, 0u, 0u, 0u};
        *(LDS_AS u32x4*)(cor + lane * 32 + 16) = u32x4{0u, 0u, 0u, 0u};
        unsigned H[8];
        decode_hi(w2, B * 0x01010101u, H);
#pragma unroll
        for (int p = 0; p < 4; ++p) { H[p] -= c0[p]; H[4 + p] -= c1[p]; }
        const unsigned o80 = (meta & 0x80u) ? 0x00800080u : 0u;
#pragma unroll
        for (int k = 0; k < 4; ++k) mma1(wf[k][0], wf[k][1], expand(H, w0, w1, k, o80));
        fv = fnext;
      }
    }
  }
  if (tv == call) bad = true;
  {
    LDS_AS f32x4* xch = (LDS_AS f32x4*)(lds + LDS_COR);
    xch[wave * 128 + lane] = h ? acc0 : acc1;
    __builtin_amdgcn_s_waitcnt(0xC07F);
    if (lane == 0) __hip_atomic_store(sync + 1 + wave, bad ? 2u : 1u, __ATOMIC_RELEASE, __HIP_MEMORY_SCOPE_WORKGROUP);
    unsigned pf;
    while ((pf = __hip_atomic_load(sync + 1 + (wave ^ 1), __ATOMIC_ACQUIRE, __HIP_MEMORY_SCOPE_WORKGROUP)) == 0) {}
    if (pf == 2u) bad = true;
    const f32x4 o = xch[(wave ^ 1) * 128 + lane];
    const f32x4 y = (h ? acc1 : acc0) + o;
    u32x2 ov = {cvt2(y[0], y[1]), cvt2(y[2], y[3])};
    if (bad) ov = u32x2{0x7FC07FC0u, 0x7FC07FC0u};
    char* op = P.out + (size_t)(src_rank * 256 + t * 16 + (lane & 15)) * (NS * 2) + nt * 64 + g * 16 + h * 8;
    *(u32x2*)op = ov;
  }
}
}

extern "C" void kernel_launch(int rank, int nranks, ncclComm_t comm, hipStream_t stream,
                              void* const* d_in, const int* in_sizes, int n_in,
                              void* d_out, int out_size, const int* in_dtypes, int out_dtype,
                              void* const* d_out_all, void* const* d_ws_all, size_t ws_bytes) {
  (void)comm; (void)in_sizes; (void)n_in; (void)out_size; (void)in_dtypes; (void)out_dtype; (void)d_out_all;
  if (nranks != NR || ws_bytes < WS_NEED || rank < 0 || rank >= NR) { fprintf(stderr, "bad config\n"); return; }
  static bool init = false;
  if (!init) {
    hipError_t e = hipFuncSetAttribute((const void*)ag_gemm, hipFuncAttributeMaxDynamicSharedMemorySize, LDS_BYTES);
    if (e != hipSuccess) { fprintf(stderr, "hipFuncSetAttribute: %s\n", hipGetErrorString(e)); return; }
    init = true;
  }
  Params P;
  P.x = (const char*)d_in[0]; P.w = (const char*)d_in[1]; P.out = (char*)d_out; P.myws = (char*)d_ws_all[rank];
  for (int q = 0; q < 14; ++q) P.peer[q] = (char*)d_ws_all[(rank + 1 + q % 7) % NR];
  P.rank = rank; P.pad = 0;
  hipLaunchKernelGGL(ag_gemm, dim3(256), dim3(THREADS), LDS_BYTES, stream, P);
}
```

```cpp
#include <hip/hip_runtime.h>
#include <rccl/rccl.h>
#include <cstdio>
#include <cstdint>

typedef unsigned u32x4 __attribute__((ext_vector_type(4)));
typedef unsigned u32x2 __attribute__((ext_vector_type(2)));
typedef unsigned short u16x2 __attribute__((ext_vector_type(2)));
typedef short s16x4 __attribute__((ext_vector_type(4)));
typedef __bf16 bf16x8 __attribute__((ext_vector_type(8)));
typedef __bf16 bf16x2 __attribute__((ext_vector_type(2)));
typedef float f32x4 __attribute__((ext_vector_type(4)));
typedef float f32x2 __attribute__((ext_vector_type(2)));
#define LDS_AS __attribute__((address_space(3)))
#define GLB_AS __attribute__((address_space(1)))

#define POLL_SLEEP 1
#define FORCE_RAW 0
#define OUTST 0
#define WARM 0
#define WARM_SLEEP 0
#define DAUX 17
#define STAUX "sc0 sc1"
#define LASTWIN 1
#define ACKQ 2
#define ACKJ 7
#define WINA 12

namespace {
constexpr int NR = 8;
constexpr int NS = 256;
constexpr int NCH = 16;
constexpr unsigned PKT = 4096;
constexpr size_t SRC_BYTES = 16 * NCH * PKT;
constexpr size_t PAR_BYTES = NR * SRC_BYTES;
constexpr size_t FLAG_OFF = 2 * PAR_BYTES;
constexpr unsigned HDR = 128;
constexpr size_t FLAG_SRC = 16 * NCH * HDR;
constexpr size_t FLAG_PAR = NR * FLAG_SRC;
constexpr size_t CTR_OFF = FLAG_OFF + 2 * FLAG_PAR;
constexpr size_t TMO_OFF = CTR_OFF + 256 * 64;
constexpr size_t STAMP_OFF = FLAG_OFF + 640 * 1024;
constexpr size_t WS_NEED = FLAG_OFF + 896 * 1024;
constexpr unsigned long long DEADLINE = 20000000ull;
constexpr int LDS_W = 2048 * 64;
constexpr int LDS_SYNC = LDS_W;
constexpr int LDS_COR = LDS_W + 64;
constexpr int LDS_BYTES = LDS_COR + 16 * 1024;
constexpr int PUSH_LDS = 40 * 1024;
constexpr int LDS_HDR = 32 * 1024;
constexpr unsigned NREC0 = 22, NREC1 = 46;
constexpr int THREADS = 576;

struct Params {
  const char* x; char* myws; const char* w; char* out;
  int rank; int pad;
  char* peer[14];
};

__device__ __forceinline__ f32x2 gelu2(f32x2 y) {
  const float c2 = -2.0f * 0.7978845608028654f * 1.4426950408889634f;
  const f32x2 k1 = {c2, c2}, k3 = {c2 * 0.044715f, c2 * 0.044715f}, one = {1.0f, 1.0f};
  const f32x2 u = y * (k1 + k3 * (y * y));
  const f32x2 d = one + f32x2{__builtin_amdgcn_exp2f(u[0]), __builtin_amdgcn_exp2f(u[1])};
  return y * f32x2{__builtin_amdgcn_rcpf(d[0]), __builtin_amdgcn_rcpf(d[1])};
}
__device__ __forceinline__ unsigned perm(unsigned s0, unsigned s1, unsigned sel) { return __builtin_amdgcn_perm(s0, s1, sel); }
__device__ __forceinline__ unsigned pkmax(unsigned a, unsigned b) {
  return __builtin_bit_cast(unsigned, __builtin_elementwise_max(__builtin_bit_cast(u16x2, a), __builtin_bit_cast(u16x2, b)));
}
__device__ __forceinline__ unsigned wave_umax(unsigned v) {
  v = max(v, (unsigned)__builtin_amdgcn_update_dpp(0, (int)v, 0x111, 0xf, 0xf, false));
  v = max(v, (unsigned)__builtin_amdgcn_update_dpp(0, (int)v, 0x112, 0xf, 0xf, false));
  v = max(v, (unsigned)__builtin_amdgcn_update_dpp(0, (int)v, 0x114, 0xf, 0xf, false));
  v = max(v, (unsigned)__builtin_amdgcn_update_dpp(0, (int)v, 0x118, 0xf, 0xf, false));
  v = max(v, (unsigned)__builtin_amdgcn_update_dpp(0, (int)v, 0x142, 0xa, 0xf, false));
  v = max(v, (unsigned)__builtin_amdgcn_update_dpp(0, (int)v, 0x143, 0xc, 0xf, false));
  return (unsigned)__builtin_amdgcn_readlane((int)v, 63);
}

typedef unsigned u32x3 __attribute__((ext_vector_type(3)));
__device__ __forceinline__ void decode_hi(const u32x3& w2, unsigned B4, unsigned (&H)[8]) {
#pragma unroll
  for (int k = 0; k < 3; ++k) {
    const unsigned n = w2[k];
    H[2 * k] = B4 + (n & 0x83838383u);
    H[2 * k + 1] = B4 + (__builtin_amdgcn_alignbit(n, n, 4) & 0x83838383u);
  }
  H[6] = B4 + (((w2[0] >> 2) & 0x01010101u) | ((w2[1] >> 1) & 0x02020202u) | ((w2[2] << 5) & 0x80808080u));
  H[7] = B4 + (((w2[0] >> 6) & 0x01010101u) | ((w2[1] >> 5) & 0x02020202u) | ((w2[2] << 1) & 0x80808080u));
}
__device__ __forceinline__ u32x4 expand(const unsigned (&H)[8], const u32x4& w0, const u32x4& w1, int f, unsigned o80) {
  const unsigned La = f < 2 ? w0[(f & 1) * 2] : w1[(f & 1) * 2], Lb = f < 2 ? w0[(f & 1) * 2 + 1] : w1[(f & 1) * 2 + 1];
  return u32x4{perm(H[2 * f], La, 0x05010400u) - o80, perm(H[2 * f], La, 0x07030602u) - o80,
               perm(H[2 * f + 1], Lb, 0x05010400u) - o80, perm(H[2 * f + 1], Lb, 0x07030602u) - o80};
}

#define STORE16(voff, data, sbase, imm) \
  asm volatile("global_store_dwordx4 %0, %1, %2 offset:%c3 " STAUX "\n\ts_nop 1" :: "v"(voff), "v"(data), "s"(sbase), "i"(imm) : "memory")
#define STORE12(voff, data, sbase, imm) \
  asm volatile("global_store_dwordx3 %0, %1, %2 offset:%c3 " STAUX "\n\ts_nop 1" :: "v"(voff), "v"(data), "s"(sbase), "i"(imm) : "memory")
#define STAMP_PARAM
#define STAMP_ARG
#define WAITV(n) asm volatile("s_waitcnt vmcnt(%c0)" :: "i"(n) : "memory")

#define GLDS4(srcp, ldsb) do { _Pragma("unroll") for (int f_ = 0; f_ < 4; ++f_) { unsigned keep_; \
      asm volatile("s_mov_b32 %0, m0\n\ts_add_u32 m0, %2, %c3\n\ts_nop 0\n\tglobal_load_lds_dwordx4 %1, off offset:%c4\n\ts_mov_b32 m0, %0" \
                   : "=&s"(keep_) : "v"(srcp), "s"(ldsb), "i"(960 * f_), "i"(64 * f_) : "memory", "scc"); } } while (0)

struct Window { unsigned o, B; };
__device__ __forceinline__ Window pick_window(const unsigned (&D)[16]) {
  const unsigned mx = pkmax(pkmax(D[0] & 0x7FFF7FFFu, D[1] & 0x7FFF7FFFu), pkmax(D[2] & 0x7FFF7FFFu, D[3] & 0x7FFF7FFFu));
  const unsigned el = max(mx >> 16, mx & 0xFFFFu) >> 7;
  const unsigned emax = wave_umax(el);
  const unsigned ntop = (unsigned)__builtin_popcountll(__ballot(el == emax));
  const unsigned etop = (ntop <= 3u && emax > 0u) ? emax - 1u : emax;
  Window w;
  w.o = (etop & 1u) ^ 1u;
  const unsigned Et = (etop + w.o) >> 1;
  w.B = Et > 3u ? Et - 3u : 0u;
  return w;
}
__device__ __forceinline__ void read_packet(const LDS_AS char* lds, int j, int lane, unsigned (&D)[16]) {
#pragma unroll
  for (int f = 0; f < 4; ++f) {
    const u32x4 v = *(const LDS_AS u32x4*)(lds + j * 4096 + f * 1024 + lane * 16);
    D[4 * f] = v[0]; D[4 * f + 1] = v[1]; D[4 * f + 2] = v[2]; D[4 * f + 3] = v[3];
  }
}

template <bool HI>
__device__ __forceinline__ void push(const Params& P, LDS_AS char* lds, volatile LDS_AS unsigned* ack, unsigned* ctrp, int pidx, int lane STAMP_PARAM) {
    const int t = pidx >> 1, h = pidx & 1;
    unsigned cv;
    asm volatile("global_load_dword %0, %1, off sc0 sc1" : "=v"(cv) : "v"(ctrp) : "memory");
    const char* src = P.x + (size_t)(t * 16 + (lane & 15)) * 4096 + h * 256 + (lane >> 4) * 16;
    const unsigned lb = (unsigned)(size_t)lds;
    GLDS4(src, lb);
    GLDS4(src + 512, lb + 4096);
    if (!HI && lane == 0) *ack = 0;
    __builtin_amdgcn_s_waitcnt(0xC07F);
    __builtin_amdgcn_s_barrier();
    const size_t doff = (size_t)P.rank * SRC_BYTES + (size_t)t * (NCH * PKT) + h * PKT;
    char* sb[7];
    { const int rot = pidx % 7;
#pragma unroll
      for (int q = 0; q < 7; ++q) { sb[q] = P.peer[rot + q]; asm volatile("" : "+s"(sb[q])); } }
    char* fp = sb[0];
    if (HI) {
#pragma unroll
      for (int q = 1; q < 7; ++q) fp = (lane >> 3) == q ? sb[q] : fp;
      fp += FLAG_OFF + (size_t)P.rank * FLAG_SRC + t * (NCH * HDR) + h * HDR + (lane & 7) * 16;
      asm volatile("" : "+v"(fp));
    }
#pragma unroll
    for (int q = 0; q < 7; ++q) { sb[q] += doff; asm volatile("" : "+s"(sb[q])); }
    asm volatile("s_waitcnt vmcnt(4)" : "+v"(cv) :: "memory");
    const unsigned call = __builtin_amdgcn_readfirstlane(cv) + 1;
    if (HI) {
      const unsigned cn = call;
      if (lane == 0) asm volatile("global_store_dword %0, %1, off sc0 sc1\n\ts_nop 1" :: "v"(ctrp), "v"(cn) : "memory");
    }
    const unsigned par = call & 1;
    const unsigned vbase = lane * 16 + par * (unsigned)PAR_BYTES;

    if (!HI) {
#pragma unroll 1
      for (int j = 0; j < 8; ++j) {
        if (j >= 1 && j < 7) GLDS4(src + (j + 1) * 512, lb + (j + 1) * 4096);
        if (j == 1) WAITV(18);
        unsigned D[16];
        read_packet(lds, j, lane, D);
        const Window wd = pick_window(D);
        const unsigned o80 = wd.o * 0x00800080u, voff = vbase + j * (2 * PKT);
        u32x4 w0, w1;
#pragma unroll
        for (int p = 0; p < 4; ++p) {
          w0[p] = perm(D[2 * p + 1] + o80, D[2 * p] + o80, 0x06040200u);
          w1[p] = perm(D[2 * p + 9] + o80, D[2 * p + 8] + o80, 0x06040200u);
        }
        if (j >= 1 && (j < 7 || LASTWIN)) {
#pragma unroll
          for (int q = 0; q < 7; ++q) {
            WAITV(WINA);
            if (q == 6 && j < ACKJ) { WAITV(12); if (lane == 0) *ack = j; }
            if (q == ACKQ && j >= ACKJ) { WAITV(2 * ACKQ); if (lane == 0) *ack = j; }
            STORE16(voff, w0, sb[q], 0); STORE16(voff, w1, sb[q], 1024);
          }
        } else {
#pragma unroll
          for (int q = 0; q < 7; ++q) { STORE16(voff, w0, sb[q], 0); STORE16(voff, w1, sb[q], 1024); }
          if (j == 7) { WAITV(14); if (lane == 0) *ack = 7; }
        }
      }
      WAITV(0);
      if (lane == 0) *ack = 8;
      { const unsigned cn = call; if (lane == 0) asm volatile("global_store_dword %0, %1, off sc0 sc1\n\ts_nop 1" :: "v"(ctrp), "v"(cn) : "memory"); }
      return;
    }

    fp += par * FLAG_PAR;
    LDS_AS char* hdr = lds + LDS_HDR;
    unsigned nprev = 0;
    auto wait_ack = [&](unsigned v) { while (*ack < v) {} };
    auto build_header = [&](int jj) -> u32x4 {
      const LDS_AS unsigned* img = (const LDS_AS unsigned*)(hdr + (jj & 1) * 128) + 3 * (lane & 7);
      u32x4 v = {call, img[0], img[1], img[2]};
      asm volatile("" : "+v"(v));
      return v;
    };
    auto send_header = [&](int jj, unsigned nrec, const u32x4& v) {
      char* fa = fp + jj * (2 * HDR);
      if (lane < 56 && (lane & 7) < (nrec > NREC0 ? 8 : 4))
        asm volatile("global_store_dwordx4 %0, %1, off sc0 sc1\n\ts_nop 1" :: "v"(fa), "v"(v) : "memory");
    };
#pragma unroll 1
    for (int j = 0; j < 8; ++j) {
      if (j >= 1 && j < 7) GLDS4(src + (j + 1) * 512, lb + (j + 1) * 4096);
      if (j == 1) WAITV(12);
      unsigned D[16];
      read_packet(lds, j, lane, D);
      const Window wd = pick_window(D);
      const unsigned B4 = wd.B * 0x01010101u, o80 = wd.o * 0x00800080u;
      unsigned inf = 0;
#pragma unroll
      for (int i = 0; i < 16; ++i) inf |= ((D[i] & 0x7F807F80u) + 0x00800080u) & 0x80008000u;
      bool raw = FORCE_RAW || __ballot(inf != 0) != 0;
      const unsigned voff = vbase + j * (2 * PKT);
      unsigned n = 0;
      if (!raw) {
        unsigned U[8], c[8], em = 0;
#pragma unroll
        for (int p = 0; p < 8; ++p) {
          const unsigned H = perm(D[2 * p + 1] + o80, D[2 * p] + o80, 0x07050301u);
          const unsigned u = (H | 0x80808080u) - B4;
          const unsigned sm = (u & 0x7C7C7C7Cu) + 0x7C7C7C7Cu;
          em |= ((~u | sm) & 0x80808080u) >> (7 - p);
          c[p] = (H & 0x80808080u) | (u & 0x03030303u);
          U[p] = u;
        }
        LDS_AS char* lst = hdr + (j & 1) * 128;
        unsigned long long bm = __ballot(em != 0);
        if (bm) {
          unsigned ovm = 0;
          LDS_AS char* dummy = hdr + 256 + lane * 2;
          do {
            const bool has = em != 0;
            const unsigned sp = (unsigned)__builtin_ctz(em | 0x80000000u);
            const bool s1 = sp & 1u, s2 = sp & 2u, s4 = sp & 4u;
            const unsigned ua = s1 ? U[1] : U[0], ub = s1 ? U[3] : U[2], uc = s1 ? U[5] : U[4], ud = s1 ? U[7] : U[6];
            const unsigned ue = s2 ? ub : ua, uf = s2 ? ud : uc;
            const unsigned code = 40u - __builtin_amdgcn_ubfe(s4 ? uf : ue, (sp & 24u) | 2u, 6u);
            ovm = max(ovm, has ? code : 0u);
            const unsigned rec = (unsigned)lane | ((sp & 7u) << 8) | ((sp >> 3) << 6) | (code << 11);
            const unsigned pos = n + __builtin_amdgcn_mbcnt_hi((unsigned)(bm >> 32), __builtin_amdgcn_mbcnt_lo((unsigned)bm, 0u));
            LDS_AS char* dst = (has && pos < NREC1) ? lst + 4 + 2 * pos : dummy;
            *(LDS_AS unsigned short*)dst = (unsigned short)rec;
            n += (unsigned)__builtin_popcountll(bm);
            em &= em - 1;
            bm = __ballot(em != 0);
          } while (bm && n <= NREC1);
          if (__ballot(ovm > 31u)) raw = true;
        }
        if (n > NREC1) raw = true;
        if (!raw) {
          u32x3 w2;
          w2[0] = c[0] | __builtin_amdgcn_alignbit(c[1], c[1], 28) | ((c[6] & 0x01010101u) << 2) | ((c[7] & 0x01010101u) << 6);
          w2[1] = c[2] | __builtin_amdgcn_alignbit(c[3], c[3], 28) | ((c[6] & 0x02020202u) << 1) | ((c[7] & 0x02020202u) << 5);
          w2[2] = c[4] | __builtin_amdgcn_alignbit(c[5], c[5], 28) | ((c[6] & 0x80808080u) >> 5) | ((c[7] & 0x80808080u) >> 1);
          if (lane == 0) *(LDS_AS unsigned*)lst = wd.B | (wd.o << 7) | (n << 16);
          const unsigned voff12 = voff - lane * 4;
#pragma unroll
          for (int q = 0; q < 7; ++q) STORE12(voff12, w2, sb[q], 2048);
        }
      }
      if (raw) {
        wait_ack(j + 1);
        n = 0;
        if (lane == 0) *(LDS_AS unsigned*)(hdr + (j & 1) * 128) = 0x100u;
        const u32x4 r0 = {D[0], D[1], D[2], D[3]}, r1 = {D[4], D[5], D[6], D[7]}, r2 = {D[8], D[9], D[10], D[11]}, r3 = {D[12], D[13], D[14], D[15]};
#pragma unroll
        for (int q = 0; q < 7; ++q) { STORE16(voff, r0, sb[q], 0); STORE16(voff, r1, sb[q], 1024); STORE16(voff, r2, sb[q], 2048); STORE16(voff, r3, sb[q], 3072); }
      }
      if (j >= 1) {
        const u32x4 hv = build_header(j - 1);
        WAITV(7);
        wait_ack(j);
        send_header(j - 1, nprev, hv);
      }
      nprev = n;
    }
    const u32x4 hv7 = build_header(7);
    WAITV(1);
    wait_ack(8);
    send_header(7, nprev, hv7);
}

__global__ void __launch_bounds__(THREADS, 3) ag_gemm(Params P) {
  extern __shared__ __attribute__((aligned(16))) char lds[];
  asm volatile("" :: "s"(P.x), "s"(P.myws));
  const int tid = threadIdx.x;
  const int lane = tid & 63;
  const int wave = __builtin_amdgcn_readfirstlane(tid >> 6);
  const int b = blockIdx.x, xcd = b & 7, bi = b >> 3;
  const bool is_push = bi < 4;
  unsigned* ctrp = (unsigned*)(P.myws + CTR_OFF + b * 64 + wave * 4);

  if (__builtin_expect(is_push, 1)) {
    const int pidx = xcd * 4 + bi;
    volatile LDS_AS unsigned* ack = (volatile LDS_AS unsigned*)((LDS_AS char*)lds + 2 * PUSH_LDS);
    if (__builtin_expect(wave == 0, 1)) push<false>(P, (LDS_AS char*)lds, ack, ctrp, pidx, lane STAMP_ARG);
    else if (wave == 1) push<true>(P, (LDS_AS char*)lds + PUSH_LDS, ack, ctrp, pidx, lane STAMP_ARG);
    return;
  }

  const int ci = xcd * 28 + (bi - 4);
  LDS_AS unsigned* sync = (LDS_AS unsigned*)(lds + LDS_SYNC);
  if (tid < 16) sync[tid] = 0;
  __builtin_amdgcn_s_waitcnt(0xC07F);
  __builtin_amdgcn_s_barrier();
  const int grp = ci >> 3, nt = ci & 7;
  const bool warm_cu = WARM && (nt & 3) == 0;
  if (wave == 8 && grp >= 16 && !warm_cu) return;
  unsigned cv = __hip_atomic_load(ctrp, __ATOMIC_RELAXED, __HIP_MEMORY_SCOPE_SYSTEM);
  if (wave < 8) {
    const char* wsrc = P.w + (size_t)(wave * 256 + (lane >> 2)) * 512 + nt * 64 + (lane & 3) * 16;
#pragma unroll 8
    for (int i = 0; i < 16; ++i)
      __builtin_amdgcn_global_load_lds((const GLB_AS void*)(wsrc + (size_t)i * 16 * 512), (LDS_AS void*)(lds + (wave * 16 + i) * 1024), 16, 0, 0);
  }
  const unsigned call = __builtin_amdgcn_readfirstlane(cv) + 1;
  if (lane == 0) __hip_atomic_store(ctrp, call, __ATOMIC_RELAXED, __HIP_MEMORY_SCOPE_SYSTEM);
  __builtin_amdgcn_s_waitcnt(0x0F70);
  asm volatile("s_waitcnt vmcnt(0)" ::: "memory");
  if (wave < 8 && lane == 0) __hip_atomic_fetch_add(sync, 1u, __ATOMIC_RELEASE, __HIP_MEMORY_SCOPE_WORKGROUP);
  while (__hip_atomic_load(sync, __ATOMIC_ACQUIRE, __HIP_MEMORY_SCOPE_WORKGROUP) < 8) __builtin_amdgcn_s_sleep(1);

  const unsigned par = call & 1;
  const int g = lane >> 4, qq = (lane >> 2) & 3, pp = lane & 3;
  const int lbase = (8 * g + qq) * 64 + 16 * pp;
  f32x4 acc0 = {0.f, 0.f, 0.f, 0.f}, acc1 = {0.f, 0.f, 0.f, 0.f};

  auto wfrag = [&](int c, int f, int tt) -> u32x4 {
    const char* p = lds + c * 8192 + lbase + f * 2048 + tt * 8;
    u32x2 lo = __builtin_bit_cast(u32x2, __builtin_amdgcn_ds_read_tr16_b64_v4i16((LDS_AS s16x4*)(p)));
    u32x2 hi = __builtin_bit_cast(u32x2, __builtin_amdgcn_ds_read_tr16_b64_v4i16((LDS_AS s16x4*)(p + 256)));
    return u32x4{lo[0], lo[1], hi[0], hi[1]};
  };
  auto mma1 = [&](const u32x4& wa_, const u32x4& wb_, const u32x4& x) {
    const bf16x8 xb = __builtin_bit_cast(bf16x8, x);
    acc0 = __builtin_amdgcn_mfma_f32_16x16x32_bf16(__builtin_bit_cast(bf16x8, wa_), xb, acc0, 0, 0, 0);
    acc1 = __builtin_amdgcn_mfma_f32_16x16x32_bf16(__builtin_bit_cast(bf16x8, wb_), xb, acc1, 0, 0, 0);
  };
  auto cvt2 = [&](float a, float c) -> unsigned {
    const f32x2 pr = gelu2(f32x2{a, c});
    return __builtin_bit_cast(unsigned, __builtin_convertvector(pr, bf16x2));
  };

  if (wave == 8) {
    if (grp < 16) {
      const int t = grp;
      const char* xs = P.x + (size_t)(t * 16 + (lane & 15)) * 4096 + (lane >> 4) * 16;
#pragma unroll 2
      for (int c = 0; c < NCH; ++c) {
        u32x4 xv[4];
#pragma unroll
        for (int f = 0; f < 4; ++f) xv[f] = *(const u32x4*)(xs + c * 256 + f * 64);
#pragma unroll
        for (int f = 0; f < 4; ++f) mma1(wfrag(c, f, 0), wfrag(c, f, 1), xv[f]);
      }
      const u32x4 ov = {cvt2(acc0[0], acc0[1]), cvt2(acc0[2], acc0[3]), cvt2(acc1[0], acc1[1]), cvt2(acc1[2], acc1[3])};
      *(u32x4*)(P.out + (size_t)(P.rank * 256 + t * 16 + (lane & 15)) * (NS * 2) + nt * 64 + g * 16) = ov;
    }
    if (warm_cu) {
      const int ps = (P.rank + 1 + (grp >> 2)) & 7, t0_ = (grp & 3) * 4;
      char* flp = P.myws + FLAG_OFF + par * FLAG_PAR + (size_t)ps * FLAG_SRC + t0_ * (NCH * HDR);
      const __amdgpu_buffer_rsrc_t frs = __builtin_amdgcn_make_buffer_rsrc(flp, 0, 4 * NCH * 16, 0x00020000);
      const int l16 = lane & 15;
      const int foff = l16 < 8 ? (l16 >> 1) * 256 + (l16 & 1) * 128 : ((l16 - 8) >> 1) * 256 + (14 + (l16 & 1)) * 16;
      const unsigned long long tq0 = __builtin_amdgcn_s_memrealtime();
      for (unsigned spin = 1;; ++spin) {
        const unsigned v = __builtin_amdgcn_raw_buffer_load_b32(frs, foff, 0, 17);
        if ((__ballot(v == call) & 0xFF00) == 0xFF00) break;
        if ((spin & 255u) == 0 && __builtin_amdgcn_s_memrealtime() - tq0 > DEADLINE) break;
      }
    }
    return;
  }

  const int h = wave & 1;
  const int src_rank = (P.rank + 1 + (grp >> 2)) & 7, t = (grp & 3) * 4 + (wave >> 1);
  unsigned* tmo = (unsigned*)(P.myws + TMO_OFF);
  bool bad = false;
  unsigned tv = 0;
  unsigned pst = 0;
  f32x4 pa0 = {0.f, 0.f, 0.f, 0.f}, pa1 = {0.f, 0.f, 0.f, 0.f};
  const LDS_AS f32x4* other = (const LDS_AS f32x4*)(lds + LDS_COR + (wave ^ 1) * 2048 + lane * 32);
  char* op = P.out + (size_t)(src_rank * 256 + t * 16 + (lane & 15)) * (NS * 2) + nt * 64 + g * 16;
  asm volatile("" : "+v"(op));
  {
    char* slot = P.myws + par * PAR_BYTES + (size_t)src_rank * SRC_BYTES + (size_t)t * (NCH * PKT) + h * PKT;
    const __amdgpu_buffer_rsrc_t drs = __builtin_amdgcn_make_buffer_rsrc(slot, 0, NCH * PKT - h * PKT, 0x00020000);
    char* flp = P.myws + FLAG_OFF + par * FLAG_PAR + (size_t)src_rank * FLAG_SRC + t * (NCH * HDR) + h * HDR;
    const __amdgpu_buffer_rsrc_t frs = __builtin_amdgcn_make_buffer_rsrc(flp, 0, NCH * HDR - h * HDR, 0x00020000);
    const int foff = (lane & 31) * 4;
    const unsigned rec0 = (lane < 32 && (lane & 3) != 0 && lane != 1) ? 2u * (3u * (lane >> 2) + (lane & 3) - 1u) - 2u : 255u;
    LDS_AS char* cor = (LDS_AS char*)lds + LDS_COR + wave * 2048;
    *(LDS_AS u32x4*)(cor + lane * 32) = u32x4{0u, 0u, 0u, 0u};
    *(LDS_AS u32x4*)(cor + lane * 32 + 16) = u32x4{0u, 0u, 0u, 0u};
    const unsigned long long t0 = __builtin_amdgcn_s_memrealtime();
    unsigned fv = __builtin_amdgcn_raw_buffer_load_b32(frs, foff, 0, 17);
#pragma unroll 1
    for (int j = 0; j < 8; ++j) {
      const int c = 2 * j + h;
      u32x4 wf[4][2];
#pragma unroll
      for (int k = 0; k < 4; ++k) { wf[k][0] = wfrag(c, k, 0); wf[k][1] = wfrag(c, k, 1); }
#pragma unroll
      for (int k = 0; k < 4; ++k) asm volatile("" : "+v"(wf[k][0]), "+v"(wf[k][1]));
      unsigned meta;
      for (unsigned spin = 1;; ++spin) {
        const unsigned long long m = __ballot(fv == call);
        meta = (unsigned)__builtin_amdgcn_readlane((int)fv, 1);
        if ((m & 0x1111u) == 0x1111u && (((meta >> 16) & 63u) <= NREC0 || (m & 0x11110000u) == 0x11110000u)) break;
        if ((spin & 1023u) == 0) {
          if (__hip_atomic_load(tmo, __ATOMIC_RELAXED, __HIP_MEMORY_SCOPE_AGENT) == call ||
              __builtin_amdgcn_s_memrealtime() - t0 > DEADLINE) {
            __hip_atomic_store(tmo, call, __ATOMIC_RELAXED, __HIP_MEMORY_SCOPE_AGENT);
            bad = true; break;
          }
        }
        __builtin_amdgcn_s_sleep(POLL_SLEEP);
        fv = __builtin_amdgcn_raw_buffer_load_b32(frs, foff, j * (int)(2 * HDR), 17);
      }
      if (bad) break;
      const int so = j * 2 * (int)PKT;
      if (__builtin_expect(meta & 0x100u, 0)) {
        const u32x4 r0 = __builtin_amdgcn_raw_buffer_load_b128(drs, lane * 16, so, DAUX);
        const u32x4 r1 = __builtin_amdgcn_raw_buffer_load_b128(drs, lane * 16 + 1024, so, DAUX);
        const u32x4 r2 = __builtin_amdgcn_raw_buffer_load_b128(drs, lane * 16 + 2048, so, DAUX);
        const u32x4 r3 = __builtin_amdgcn_raw_buffer_load_b128(drs, lane * 16 + 3072, so, DAUX);
        fv = __builtin_amdgcn_raw_buffer_load_b32(frs, foff, ((j + 1) & 7) * (int)(2 * HDR), 17);
        tv = __hip_atomic_load(tmo, __ATOMIC_RELAXED, __HIP_MEMORY_SCOPE_AGENT);
        mma1(wf[0][0], wf[0][1], r0); mma1(wf[1][0], wf[1][1], r1); mma1(wf[2][0], wf[2][1], r2); mma1(wf[3][0], wf[3][1], r3);
      } else {
        const unsigned hv = fv;
        asm volatile("" :: "s"(meta), "v"(hv) : "memory");
        const u32x3 w2 = __builtin_amdgcn_raw_buffer_load_b96(drs, lane * 12 + 2048, so, DAUX);
        const u32x4 w0 = __builtin_amdgcn_raw_buffer_load_b128(drs, lane * 16, so, DAUX);
        const u32x4 w1 = __builtin_amdgcn_raw_buffer_load_b128(drs, lane * 16 + 1024, so, DAUX);
        const unsigned fnext = __builtin_amdgcn_raw_buffer_load_b32(frs, foff, ((j + 1) & 7) * (int)(2 * HDR), 17);
        tv = __hip_atomic_load(tmo, __ATOMIC_RELAXED, __HIP_MEMORY_SCOPE_AGENT);
        const unsigned B = meta & 0x7Fu, nrec = (meta >> 16) & 63u;
#pragma unroll
        for (int e = 0; e < 2; ++e) {
          const unsigned rec = e ? hv >> 16 : hv & 0xFFFFu;
          const unsigned slot = (rec >> 6) & 31u;
          const unsigned d = ((rec >> 11) - 8u) * 4u;
          if (rec0 + e < nrec)
            __hip_atomic_fetch_add((LDS_AS unsigned*)(cor + (rec & 63u) * 32 + (slot >> 2) * 4), d << (8 * (slot & 3u)), __ATOMIC_RELAXED, __HIP_MEMORY_SCOPE_WORKGROUP);
        }
        asm volatile("" ::: "memory");
        const u32x4 c0 = *(LDS_AS u32x4*)(cor + lane * 32), c1 = *(LDS_AS u32x4*)(cor + lane * 32 + 16);
        asm volatile("" ::: "memory");
        *(LDS_AS u32x4*)(cor + lane * 32) = u32x4{0u, 0u, 0u, 0u};
        *(LDS_AS u32x4*)(cor + lane * 32 + 16) = u32x4{0u, 0u, 0u, 0u};
        if (j == 7) {
          pst = __builtin_amdgcn_readfirstlane(*(volatile LDS_AS unsigned*)(sync + 1 + (wave >> 1)));
          asm volatile("" ::: "memory");
          if (pst & 1u) { pa0 = other[0]; pa1 = other[1]; }
        }
        unsigned H[8];
        decode_hi(w2, B * 0x01010101u, H);
#pragma unroll
        for (int p = 0; p < 4; ++p) { H[p] -= c0[p]; H[4 + p] -= c1[p]; }
        const unsigned o80 = (meta & 0x80u) ? 0x00800080u : 0u;
#pragma unroll
        for (int k = 0; k < 4; ++k) mma1(wf[k][0], wf[k][1], expand(H, w0, w1, k, o80));
        fv = fnext;
      }
    }
  }
  if (tv == call) bad = true;
  if (!(pst & 1u)) {
    LDS_AS f32x4* mine = (LDS_AS f32x4*)(lds + LDS_COR + wave * 2048 + lane * 32);
    mine[0] = acc0; mine[1] = acc1;
    __builtin_amdgcn_s_waitcnt(0xC07F);
    unsigned old = 0;
    if (lane == 0) old = __hip_atomic_fetch_add(sync + 1 + (wave >> 1), bad ? 3u : 1u, __ATOMIC_ACQ_REL, __HIP_MEMORY_SCOPE_WORKGROUP);
    pst = __builtin_amdgcn_readfirstlane(old);
    if (pst & 1u) { pa0 = other[0]; pa1 = other[1]; }
  }
  if (pst & 1u) {
    if (pst & 2u) bad = true;
    const f32x4 y0 = acc0 + pa0, y1 = acc1 + pa1;
    u32x4 ov = {cvt2(y0[0], y0[1]), cvt2(y0[2], y0[3]), cvt2(y1[0], y1[1]), cvt2(y1[2], y1[3])};
    if (bad) ov = u32x4{0x7FC07FC0u, 0x7FC07FC0u, 0x7FC07FC0u, 0x7FC07FC0u};
    *(u32x4*)op = ov;
  }
}
}

extern "C" void kernel_launch(int rank, int nranks, ncclComm_t comm, hipStream_t stream,
                              void* const* d_in, const int* in_sizes, int n_in,
                              void* d_out, int out_size, const int* in_dtypes, int out_dtype,
                              void* const* d_out_all, void* const* d_ws_all, size_t ws_bytes) {
  (void)comm; (void)in_sizes; (void)n_in; (void)out_size; (void)in_dtypes; (void)out_dtype; (void)d_out_all;
  if (nranks != NR || ws_bytes < WS_NEED || rank < 0 || rank >= NR) { fprintf(stderr, "bad config\n"); return; }
  static bool init = false;
  if (!init) {
    hipError_t e = hipFuncSetAttribute((const void*)ag_gemm, hipFuncAttributeMaxDynamicSharedMemorySize, LDS_BYTES);
    if (e != hipSuccess) { fprintf(stderr, "hipFuncSetAttribute: %s\n", hipGetErrorString(e)); return; }
    init = true;
  }
  Params P;
  P.x = (const char*)d_in[0]; P.w = (const char*)d_in[1]; P.out = (char*)d_out; P.myws = (char*)d_ws_all[rank];
  for (int q = 0; q < 14; ++q) P.peer[q] = (char*)d_ws_all[(rank + 1 + q % 7) % NR];
  P.rank = rank; P.pad = 0;
  hipLaunchKernelGGL(ag_gemm, dim3(256), dim3(THREADS), LDS_BYTES, stream, P);
}
```

```cpp
#include <hip/hip_runtime.h>
#include <rccl/rccl.h>
#include <cstdio>
#include <cstdint>

typedef unsigned u32x4 __attribute__((ext_vector_type(4)));
typedef unsigned u32x2 __attribute__((ext_vector_type(2)));
typedef unsigned short u16x2 __attribute__((ext_vector_type(2)));
typedef short s16x4 __attribute__((ext_vector_type(4)));
typedef __bf16 bf16x8 __attribute__((ext_vector_type(8)));
typedef __bf16 bf16x2 __attribute__((ext_vector_type(2)));
typedef float f32x4 __attribute__((ext_vector_type(4)));
typedef float f32x2 __attribute__((ext_vector_type(2)));
#define LDS_AS __attribute__((address_space(3)))
#define GLB_AS __attribute__((address_space(1)))

namespace {
constexpr int NR = 8;
constexpr int NS = 256;
constexpr int NCH = 16;
constexpr unsigned PKT = 4096;
constexpr size_t SRC_BYTES = 16 * NCH * PKT;
constexpr size_t PAR_BYTES = NR * SRC_BYTES;
constexpr size_t FLAG_OFF = 2 * PAR_BYTES;
constexpr unsigned HDR = 128;
constexpr size_t FLAG_SRC = 16 * NCH * HDR;
constexpr size_t FLAG_PAR = NR * FLAG_SRC;
constexpr size_t CTR_OFF = FLAG_OFF + 2 * FLAG_PAR;
constexpr size_t TMO_OFF = CTR_OFF + 256 * 64;
constexpr size_t WS_NEED = TMO_OFF + 128;
constexpr unsigned long long DEADLINE = 20000000ull;
constexpr int LDS_W = 2048 * 64;
constexpr int LDS_SYNC = LDS_W;
constexpr int LDS_COR = LDS_W + 64;
constexpr int LDS_BYTES = LDS_COR + 16 * 1024;
constexpr int PUSH_LDS = 40 * 1024;
constexpr int LDS_HDR = 32 * 1024;
constexpr int WIN_LO = 12;
constexpr unsigned NREC0 = 22, NREC1 = 46;
constexpr int THREADS = 576;

struct Params {
  const char* x; char* myws; const char* w; char* out;
  int rank; int pad;
  char* peer[14];
};

__device__ __forceinline__ f32x2 gelu2(f32x2 y) {
  const float c2 = -2.0f * 0.7978845608028654f * 1.4426950408889634f;
  const f32x2 k1 = {c2, c2}, k3 = {c2 * 0.044715f, c2 * 0.044715f}, one = {1.0f, 1.0f};
  const f32x2 u = y * (k1 + k3 * (y * y));
  const f32x2 d = one + f32x2{__builtin_amdgcn_exp2f(u[0]), __builtin_amdgcn_exp2f(u[1])};
  return y * f32x2{__builtin_amdgcn_rcpf(d[0]), __builtin_amdgcn_rcpf(d[1])};
}
__device__ __forceinline__ unsigned perm(unsigned s0, unsigned s1, unsigned sel) { return __builtin_amdgcn_perm(s0, s1, sel); }
__device__ __forceinline__ unsigned pkmax(unsigned a, unsigned b) {
  return __builtin_bit_cast(unsigned, __builtin_elementwise_max(__builtin_bit_cast(u16x2, a), __builtin_bit_cast(u16x2, b)));
}
__device__ __forceinline__ unsigned wave_umax(unsigned v) {
  v = max(v, (unsigned)__builtin_amdgcn_update_dpp(0, (int)v, 0x111, 0xf, 0xf, false));
  v = max(v, (unsigned)__builtin_amdgcn_update_dpp(0, (int)v, 0x112, 0xf, 0xf, false));
  v = max(v, (unsigned)__builtin_amdgcn_update_dpp(0, (int)v, 0x114, 0xf, 0xf, false));
  v = max(v, (unsigned)__builtin_amdgcn_update_dpp(0, (int)v, 0x118, 0xf, 0xf, false));
  v = max(v, (unsigned)__builtin_amdgcn_update_dpp(0, (int)v, 0x142, 0xa, 0xf, false));
  v = max(v, (unsigned)__builtin_amdgcn_update_dpp(0, (int)v, 0x143, 0xc, 0xf, false));
  return (unsigned)__builtin_amdgcn_readlane((int)v, 63);
}

typedef unsigned u32x3 __attribute__((ext_vector_type(3)));
__device__ __forceinline__ void decode_hi(const u32x3& w2, unsigned B4, unsigned (&H)[8]) {
#pragma unroll
  for (int k = 0; k < 3; ++k) {
    const unsigned n = w2[k];
    H[2 * k] = B4 + (n & 0x83838383u);
    H[2 * k + 1] = B4 + (__builtin_amdgcn_alignbit(n, n, 4) & 0x83838383u);
  }
  H[6] = B4 + (((w2[0] >> 2) & 0x01010101u) | ((w2[1] >> 1) & 0x02020202u) | ((w2[2] << 5) & 0x80808080u));
  H[7] = B4 + (((w2[0] >> 6) & 0x01010101u) | ((w2[1] >> 5) & 0x02020202u) | ((w2[2] << 1) & 0x80808080u));
}
__device__ __forceinline__ u32x4 expand(const unsigned (&H)[8], const u32x4& w0, const u32x4& w1, int f, unsigned o80) {
  const unsigned La = f < 2 ? w0[(f & 1) * 2] : w1[(f & 1) * 2], Lb = f < 2 ? w0[(f & 1) * 2 + 1] : w1[(f & 1) * 2 + 1];
  return u32x4{perm(H[2 * f], La, 0x05010400u) - o80, perm(H[2 * f], La, 0x07030602u) - o80,
               perm(H[2 * f + 1], Lb, 0x05010400u) - o80, perm(H[2 * f + 1], Lb, 0x07030602u) - o80};
}

#define STORE16(voff, data, sbase, imm) \
  asm volatile("global_store_dwordx4 %0, %1, %2 offset:%c3 sc0 sc1\n\ts_nop 1" :: "v"(voff), "v"(data), "s"(sbase), "i"(imm) : "memory")
#define STORE12(voff, data, sbase, imm) \
  asm volatile("global_store_dwordx3 %0, %1, %2 offset:%c3 sc0 sc1\n\ts_nop 1" :: "v"(voff), "v"(data), "s"(sbase), "i"(imm) : "memory")
#define WAITV(n) asm volatile("s_waitcnt vmcnt(%c0)" :: "i"(n) : "memory")

#define GLDS4(srcp, ldsb) do { _Pragma("unroll") for (int f_ = 0; f_ < 4; ++f_) { unsigned keep_; \
      asm volatile("s_mov_b32 %0, m0\n\ts_add_u32 m0, %2, %c3\n\ts_nop 0\n\tglobal_load_lds_dwordx4 %1, off offset:%c4\n\ts_mov_b32 m0, %0" \
                   : "=&s"(keep_) : "v"(srcp), "s"(ldsb), "i"(960 * f_), "i"(64 * f_) : "memory", "scc"); } } while (0)

struct Window { unsigned o, B; };
__device__ __forceinline__ Window pick_window(const unsigned (&D)[16]) {
  const unsigned mx = pkmax(pkmax(D[0] & 0x7FFF7FFFu, D[1] & 0x7FFF7FFFu), pkmax(D[2] & 0x7FFF7FFFu, D[3] & 0x7FFF7FFFu));
  const unsigned el = max(mx >> 16, mx & 0xFFFFu) >> 7;
  const unsigned emax = wave_umax(el);
  const unsigned ntop = (unsigned)__builtin_popcountll(__ballot(el == emax));
  const unsigned etop = (ntop <= 3u && emax > 0u) ? emax - 1u : emax;
  Window w;
  w.o = (etop & 1u) ^ 1u;
  const unsigned Et = (etop + w.o) >> 1;
  w.B = Et > 3u ? Et - 3u : 0u;
  return w;
}
__device__ __forceinline__ void read_packet(const LDS_AS char* lds, int j, int lane, unsigned (&D)[16]) {
#pragma unroll
  for (int f = 0; f < 4; ++f) {
    const u32x4 v = *(const LDS_AS u32x4*)(lds + j * 4096 + f * 1024 + lane * 16);
    D[4 * f] = v[0]; D[4 * f + 1] = v[1]; D[4 * f + 2] = v[2]; D[4 * f + 3] = v[3];
  }
}

template <bool HI>
__device__ __forceinline__ void push(const Params& P, LDS_AS char* lds, volatile LDS_AS unsigned* ack, unsigned* ctrp, int pidx, int lane) {
    const int t = pidx >> 1, h = pidx & 1;
    unsigned cv;
    asm volatile("global_load_dword %0, %1, off sc0 sc1" : "=v"(cv) : "v"(ctrp) : "memory");
    const char* src = P.x + (size_t)(t * 16 + (lane & 15)) * 4096 + h * 256 + (lane >> 4) * 16;
    const unsigned lb = (unsigned)(size_t)lds;
    u32x4 R0[4] = {};
    if (!HI) {
#pragma unroll
      for (int f = 0; f < 4; ++f) asm volatile("global_load_dwordx4 %0, %1, off offset:%c2" : "=v"(R0[f]) : "v"(src), "i"(64 * f) : "memory");
    } else GLDS4(src, lb);
    GLDS4(src + 512, lb + 4096);
    if (!HI && lane == 0) *ack = 0;
    __builtin_amdgcn_s_waitcnt(0xC07F);
    __builtin_amdgcn_s_barrier();
    const size_t doff = (size_t)P.rank * SRC_BYTES + (size_t)t * (NCH * PKT) + h * PKT;
    char* sb[7];
    { const int rot = pidx % 7;
#pragma unroll
      for (int q = 0; q < 7; ++q) { sb[q] = P.peer[rot + q]; asm volatile("" : "+s"(sb[q])); } }
    char* fp = sb[0];
    if (HI) {
#pragma unroll
      for (int q = 1; q < 7; ++q) fp = (lane >> 3) == q ? sb[q] : fp;
      fp += FLAG_OFF + (size_t)P.rank * FLAG_SRC + t * (NCH * HDR) + h * HDR + (lane & 7) * 16;
      asm volatile("" : "+v"(fp));
    }
#pragma unroll
    for (int q = 0; q < 7; ++q) { sb[q] += doff; asm volatile("" : "+s"(sb[q])); }
    asm volatile("s_waitcnt vmcnt(4)" : "+v"(cv), "+v"(R0[0]), "+v"(R0[1]), "+v"(R0[2]), "+v"(R0[3]) :: "memory");
    const unsigned call = __builtin_amdgcn_readfirstlane(cv) + 1;
    if (HI) {
      const unsigned cn = call;
      if (lane == 0) asm volatile("global_store_dword %0, %1, off sc0 sc1\n\ts_nop 1" :: "v"(ctrp), "v"(cn) : "memory");
    }
    const unsigned par = call & 1;
    const unsigned vbase = lane * 16 + par * (unsigned)PAR_BYTES;

    if (!HI) {
      {
        unsigned D[16];
#pragma unroll
        for (int f = 0; f < 4; ++f) { D[4 * f] = R0[f][0]; D[4 * f + 1] = R0[f][1]; D[4 * f + 2] = R0[f][2]; D[4 * f + 3] = R0[f][3]; }
        const unsigned o80 = pick_window(D).o * 0x00800080u;
        u32x4 w0, w1;
#pragma unroll
        for (int p = 0; p < 4; ++p) w0[p] = perm(D[2 * p + 1] + o80, D[2 * p] + o80, 0x06040200u);
#pragma unroll
        for (int q = 0; q < 7; ++q) STORE16(vbase, w0, sb[q], 0);
#pragma unroll
        for (int p = 0; p < 4; ++p) w1[p] = perm(D[2 * p + 9] + o80, D[2 * p + 8] + o80, 0x06040200u);
#pragma unroll
        for (int q = 0; q < 7; ++q) STORE16(vbase, w1, sb[q], 1024);
      }
#pragma unroll 1
      for (int j = 1; j < 8; ++j) {
        if (j < 7) GLDS4(src + (j + 1) * 512, lb + (j + 1) * 4096);
        if (j == 1) WAITV(18);
        unsigned D[16];
        read_packet(lds, j, lane, D);
        const Window wd = pick_window(D);
        const unsigned o80 = wd.o * 0x00800080u, voff = vbase + j * (2 * PKT);
        u32x4 w0, w1;
#pragma unroll
        for (int p = 0; p < 4; ++p) {
          w0[p] = perm(D[2 * p + 1] + o80, D[2 * p] + o80, 0x06040200u);
          w1[p] = perm(D[2 * p + 9] + o80, D[2 * p + 8] + o80, 0x06040200u);
        }
#pragma unroll
        for (int q = 0; q < 7; ++q) {
          WAITV(WIN_LO);
          if (q == 6 && j < 7) { WAITV(12); if (lane == 0) *ack = j; }
          if (q == 2 && j == 7) { WAITV(4); if (lane == 0) *ack = 7; }
          STORE16(voff, w0, sb[q], 0); STORE16(voff, w1, sb[q], 1024);
        }
      }
      WAITV(0);
      if (lane == 0) *ack = 8;
      { const unsigned cn = call; if (lane == 0) asm volatile("global_store_dword %0, %1, off sc0 sc1\n\ts_nop 1" :: "v"(ctrp), "v"(cn) : "memory"); }
      return;
    }

    fp += par * FLAG_PAR;
    LDS_AS char* hdr = lds + LDS_HDR;
    unsigned nprev = 0;
    auto wait_ack = [&](unsigned v) { while (*ack < v) {} };
    auto build_header = [&](int jj) -> u32x4 {
      const LDS_AS unsigned* img = (const LDS_AS unsigned*)(hdr + (jj & 1) * 128) + 3 * (lane & 7);
      u32x4 v = {call, img[0], img[1], img[2]};
      asm volatile("" : "+v"(v));
      return v;
    };
    auto send_header = [&](int jj, unsigned nrec, const u32x4& v) {
      char* fa = fp + jj * (2 * HDR);
      if (lane < 56 && (lane & 7) < (nrec > NREC0 ? 8 : 4))
        asm volatile("global_store_dwordx4 %0, %1, off sc0 sc1\n\ts_nop 1" :: "v"(fa), "v"(v) : "memory");
    };
#pragma unroll 1
    for (int j = 0; j < 8; ++j) {
      if (j >= 1 && j < 7) GLDS4(src + (j + 1) * 512, lb + (j + 1) * 4096);
      if (j == 1) WAITV(12);
      unsigned D[16];
      read_packet(lds, j, lane, D);
      const Window wd = pick_window(D);
      const unsigned B4 = wd.B * 0x01010101u, o80 = wd.o * 0x00800080u;
      unsigned inf = 0;
#pragma unroll
      for (int i = 0; i < 16; ++i) inf |= ((D[i] & 0x7F807F80u) + 0x00800080u) & 0x80008000u;
      bool raw = __ballot(inf != 0) != 0;
      const unsigned voff = vbase + j * (2 * PKT);
      unsigned n = 0;
      if (!raw) {
        unsigned U[8], c[8], em = 0;
#pragma unroll
        for (int p = 0; p < 8; ++p) {
          const unsigned H = perm(D[2 * p + 1] + o80, D[2 * p] + o80, 0x07050301u);
          const unsigned u = (H | 0x80808080u) - B4;
          const unsigned sm = (u & 0x7C7C7C7Cu) + 0x7C7C7C7Cu;
          em |= ((~u | sm) & 0x80808080u) >> (7 - p);
          c[p] = (H & 0x80808080u) | (u & 0x03030303u);
          U[p] = u;
        }
        LDS_AS char* lst = hdr + (j & 1) * 128;
        unsigned long long bm = __ballot(em != 0);
        if (bm) {
          unsigned ovm = 0;
          LDS_AS char* dummy = hdr + 256 + lane * 2;
          do {
            const bool has = em != 0;
            const unsigned sp = (unsigned)__builtin_ctz(em | 0x80000000u);
            const bool s1 = sp & 1u, s2 = sp & 2u, s4 = sp & 4u;
            const unsigned ua = s1 ? U[1] : U[0], ub = s1 ? U[3] : U[2], uc = s1 ? U[5] : U[4], ud = s1 ? U[7] : U[6];
            const unsigned ue = s2 ? ub : ua, uf = s2 ? ud : uc;
            const unsigned code = 40u - __builtin_amdgcn_ubfe(s4 ? uf : ue, (sp & 24u) | 2u, 6u);
            ovm = max(ovm, has ? code : 0u);
            const unsigned rec = (unsigned)lane | ((sp & 7u) << 8) | ((sp >> 3) << 6) | (code << 11);
            const unsigned pos = n + __builtin_amdgcn_mbcnt_hi((unsigned)(bm >> 32), __builtin_amdgcn_mbcnt_lo((unsigned)bm, 0u));
            LDS_AS char* dst = (has && pos < NREC1) ? lst + 4 + 2 * pos : dummy;
            *(LDS_AS unsigned short*)dst = (unsigned short)rec;
            n += (unsigned)__builtin_popcountll(bm);
            em &= em - 1;
            bm = __ballot(em != 0);
          } while (bm && n <= NREC1);
          if (__ballot(ovm > 31u)) raw = true;
        }
        if (n > NREC1) raw = true;
        if (!raw) {
          u32x3 w2;
          w2[0] = c[0] | __builtin_amdgcn_alignbit(c[1], c[1], 28) | ((c[6] & 0x01010101u) << 2) | ((c[7] & 0x01010101u) << 6);
          w2[1] = c[2] | __builtin_amdgcn_alignbit(c[3], c[3], 28) | ((c[6] & 0x02020202u) << 1) | ((c[7] & 0x02020202u) << 5);
          w2[2] = c[4] | __builtin_amdgcn_alignbit(c[5], c[5], 28) | ((c[6] & 0x80808080u) >> 5) | ((c[7] & 0x80808080u) >> 1);
          if (lane == 0) *(LDS_AS unsigned*)lst = wd.B | (wd.o << 7) | (n << 16);
          const unsigned voff12 = voff - lane * 4;
#pragma unroll
          for (int q = 0; q < 7; ++q) STORE12(voff12, w2, sb[q], 2048);
        }
      }
      if (raw) {
        wait_ack(j + 1);
        n = 0;
        if (lane == 0) *(LDS_AS unsigned*)(hdr + (j & 1) * 128) = 0x100u;
        const u32x4 r0 = {D[0], D[1], D[2], D[3]}, r1 = {D[4], D[5], D[6], D[7]}, r2 = {D[8], D[9], D[10], D[11]}, r3 = {D[12], D[13], D[14], D[15]};
#pragma unroll
        for (int q = 0; q < 7; ++q) { STORE16(voff, r0, sb[q], 0); STORE16(voff, r1, sb[q], 1024); STORE16(voff, r2, sb[q], 2048); STORE16(voff, r3, sb[q], 3072); }
      }
      if (j >= 1) {
        const u32x4 hv = build_header(j - 1);
        WAITV(7);
        wait_ack(j);
        send_header(j - 1, nprev, hv);
      }
      nprev = n;
    }
    const u32x4 hv7 = build_header(7);
    WAITV(1);
    wait_ack(8);
    send_header(7, nprev, hv7);
}

__global__ void __launch_bounds__(THREADS, 3) ag_gemm(Params P) {
  extern __shared__ __attribute__((aligned(16))) char lds[];
  asm volatile("" :: "s"(P.x), "s"(P.myws));
  const int tid = threadIdx.x;
  const int lane = tid & 63;
  const int wave = __builtin_amdgcn_readfirstlane(tid >> 6);
  const int b = blockIdx.x, xcd = b & 7, bi = b >> 3;
  const bool is_push = bi < 4;
  unsigned* ctrp = (unsigned*)(P.myws + CTR_OFF + b * 64 + wave * 4);

  if (__builtin_expect(is_push, 1)) {
    const int pidx = xcd * 4 + bi;
    volatile LDS_AS unsigned* ack = (volatile LDS_AS unsigned*)((LDS_AS char*)lds + 2 * PUSH_LDS);
    if (__builtin_expect(wave == 0, 1)) push<false>(P, (LDS_AS char*)lds, ack, ctrp, pidx, lane);
    else if (wave == 1) push<true>(P, (LDS_AS char*)lds + PUSH_LDS, ack, ctrp, pidx, lane);
    return;
  }

  const int ci = xcd * 28 + (bi - 4);
  LDS_AS unsigned* sync = (LDS_AS unsigned*)(lds + LDS_SYNC);
  if (tid < 16) sync[tid] = 0;
  __builtin_amdgcn_s_waitcnt(0xC07F);
  __builtin_amdgcn_s_barrier();
  const int grp = ci >> 3, nt = ci & 7;
  if (wave == 8 && grp >= 16) return;
  unsigned cv = __hip_atomic_load(ctrp, __ATOMIC_RELAXED, __HIP_MEMORY_SCOPE_SYSTEM);
  if (wave < 8) {
    const char* wsrc = P.w + (size_t)(wave * 256 + (lane >> 2)) * 512 + nt * 64 + (lane & 3) * 16;
#pragma unroll 8
    for (int i = 0; i < 16; ++i)
      __builtin_amdgcn_global_load_lds((const GLB_AS void*)(wsrc + (size_t)i * 16 * 512), (LDS_AS void*)(lds + (wave * 16 + i) * 1024), 16, 0, 0);
  }
  const unsigned call = __builtin_amdgcn_readfirstlane(cv) + 1;
  if (lane == 0) __hip_atomic_store(ctrp, call, __ATOMIC_RELAXED, __HIP_MEMORY_SCOPE_SYSTEM);
  __builtin_amdgcn_s_waitcnt(0x0F70);
  asm volatile("s_waitcnt vmcnt(0)" ::: "memory");
  if (wave < 8 && lane == 0) __hip_atomic_fetch_add(sync, 1u, __ATOMIC_RELEASE, __HIP_MEMORY_SCOPE_WORKGROUP);
  while (__hip_atomic_load(sync, __ATOMIC_ACQUIRE, __HIP_MEMORY_SCOPE_WORKGROUP) < 8) __builtin_amdgcn_s_sleep(1);

  const unsigned par = call & 1;
  const int g = lane >> 4, qq = (lane >> 2) & 3, pp = lane & 3;
  const int lbase = (8 * g + qq) * 64 + 16 * pp;
  f32x4 acc0 = {0.f, 0.f, 0.f, 0.f}, acc1 = {0.f, 0.f, 0.f, 0.f};

  auto wfrag = [&](int c, int f, int tt) -> u32x4 {
    const char* p = lds + c * 8192 + lbase + f * 2048 + tt * 8;
    u32x2 lo = __builtin_bit_cast(u32x2, __builtin_amdgcn_ds_read_tr16_b64_v4i16((LDS_AS s16x4*)(p)));
    u32x2 hi = __builtin_bit_cast(u32x2, __builtin_amdgcn_ds_read_tr16_b64_v4i16((LDS_AS s16x4*)(p + 256)));
    return u32x4{lo[0], lo[1], hi[0], hi[1]};
  };
  auto mma1 = [&](const u32x4& wa_, const u32x4& wb_, const u32x4& x) {
    const bf16x8 xb = __builtin_bit_cast(bf16x8, x);
    acc0 = __builtin_amdgcn_mfma_f32_16x16x32_bf16(__builtin_bit_cast(bf16x8, wa_), xb, acc0, 0, 0, 0);
    acc1 = __builtin_amdgcn_mfma_f32_16x16x32_bf16(__builtin_bit_cast(bf16x8, wb_), xb, acc1, 0, 0, 0);
  };
  auto cvt2 = [&](float a, float c) -> unsigned {
    const f32x2 pr = gelu2(f32x2{a, c});
    return __builtin_bit_cast(unsigned, __builtin_convertvector(pr, bf16x2));
  };

  if (wave == 8) {
    const int t = grp;
    const char* xs = P.x + (size_t)(t * 16 + (lane & 15)) * 4096 + (lane >> 4) * 16;
#pragma unroll 2
    for (int c = 0; c < NCH; ++c) {
      u32x4 xv[4];
#pragma unroll
      for (int f = 0; f < 4; ++f) xv[f] = *(const u32x4*)(xs + c * 256 + f * 64);
#pragma unroll
      for (int f = 0; f < 4; ++f) mma1(wfrag(c, f, 0), wfrag(c, f, 1), xv[f]);
    }
    const u32x4 ov = {cvt2(acc0[0], acc0[1]), cvt2(acc0[2], acc0[3]), cvt2(acc1[0], acc1[1]), cvt2(acc1[2], acc1[3])};
    char* sop = P.out + (size_t)(P.rank * 256 + t * 16 + (lane & 15)) * (NS * 2) + nt * 64 + g * 16;
    asm volatile("global_store_dwordx4 %0, %1, off sc0 sc1\n\ts_nop 1" :: "v"(sop), "v"(ov) : "memory");
    return;
  }

  const int h = wave & 1;
  const int src_rank = (P.rank + 1 + (grp >> 2)) & 7, t = (grp & 3) * 4 + (wave >> 1);
  unsigned* tmo = (unsigned*)(P.myws + TMO_OFF);
  bool bad = false;
  unsigned tv = 0;
  unsigned pst = 0;
  f32x4 pa0 = {0.f, 0.f, 0.f, 0.f}, pa1 = {0.f, 0.f, 0.f, 0.f};
  const LDS_AS f32x4* other = (const LDS_AS f32x4*)(lds + LDS_COR + (wave ^ 1) * 2048 + lane * 32);
  char* op = P.out + (size_t)(src_rank * 256 + t * 16 + (lane & 15)) * (NS * 2) + nt * 64 + g * 16;
  asm volatile("" : "+v"(op));
  {
    char* slot = P.myws + par * PAR_BYTES + (size_t)src_rank * SRC_BYTES + (size_t)t * (NCH * PKT) + h * PKT;
    const __amdgpu_buffer_rsrc_t drs = __builtin_amdgcn_make_buffer_rsrc(slot, 0, NCH * PKT - h * PKT, 0x00020000);
    char* flp = P.myws + FLAG_OFF + par * FLAG_PAR + (size_t)src_rank * FLAG_SRC + t * (NCH * HDR) + h * HDR;
    const __amdgpu_buffer_rsrc_t frs = __builtin_amdgcn_make_buffer_rsrc(flp, 0, NCH * HDR - h * HDR, 0x00020000);
    const int foff = (lane & 31) * 4;
    const unsigned rec0 = (lane < 32 && (lane & 3) != 0 && lane != 1) ? 2u * (3u * (lane >> 2) + (lane & 3) - 1u) - 2u : 255u;
    LDS_AS char* cor = (LDS_AS char*)lds + LDS_COR + wave * 2048;
    *(LDS_AS u32x4*)(cor + lane * 32) = u32x4{0u, 0u, 0u, 0u};
    *(LDS_AS u32x4*)(cor + lane * 32 + 16) = u32x4{0u, 0u, 0u, 0u};
    const unsigned long long t0 = __builtin_amdgcn_s_memrealtime();
    unsigned fv = __builtin_amdgcn_raw_buffer_load_b32(frs, foff, 0, 17);
#pragma unroll 1
    for (int j = 0; j < 8; ++j) {
      const int c = 2 * j + h;
      u32x4 wf[4][2];
#pragma unroll
      for (int k = 0; k < 4; ++k) { wf[k][0] = wfrag(c, k, 0); wf[k][1] = wfrag(c, k, 1); }
#pragma unroll
      for (int k = 0; k < 4; ++k) asm volatile("" : "+v"(wf[k][0]), "+v"(wf[k][1]));
      unsigned meta;
      for (unsigned spin = 1;; ++spin) {
        const unsigned long long m = __ballot(fv == call);
        meta = (unsigned)__builtin_amdgcn_readlane((int)fv, 1);
        if ((m & 0x1111u) == 0x1111u && (((meta >> 16) & 63u) <= NREC0 || (m & 0x11110000u) == 0x11110000u)) break;
        if ((spin & 1023u) == 0) {
          if (__hip_atomic_load(tmo, __ATOMIC_RELAXED, __HIP_MEMORY_SCOPE_AGENT) == call ||
              __builtin_amdgcn_s_memrealtime() - t0 > DEADLINE) {
            __hip_atomic_store(tmo, call, __ATOMIC_RELAXED, __HIP_MEMORY_SCOPE_AGENT);
            bad = true; break;
          }
        }
        __builtin_amdgcn_s_sleep(1);
        fv = __builtin_amdgcn_raw_buffer_load_b32(frs, foff, j * (int)(2 * HDR), 17);
      }
      if (bad) break;
      const int so = j * 2 * (int)PKT;
      if (__builtin_expect(meta & 0x100u, 0)) {
        const u32x4 r0 = __builtin_amdgcn_raw_buffer_load_b128(drs, lane * 16, so, 17);
        const u32x4 r1 = __builtin_amdgcn_raw_buffer_load_b128(drs, lane * 16 + 1024, so, 17);
        const u32x4 r2 = __builtin_amdgcn_raw_buffer_load_b128(drs, lane * 16 + 2048, so, 17);
        const u32x4 r3 = __builtin_amdgcn_raw_buffer_load_b128(drs, lane * 16 + 3072, so, 17);
        fv = __builtin_amdgcn_raw_buffer_load_b32(frs, foff, ((j + 1) & 7) * (int)(2 * HDR), 17);
        tv = __hip_atomic_load(tmo, __ATOMIC_RELAXED, __HIP_MEMORY_SCOPE_AGENT);
        mma1(wf[0][0], wf[0][1], r0); mma1(wf[1][0], wf[1][1], r1); mma1(wf[2][0], wf[2][1], r2); mma1(wf[3][0], wf[3][1], r3);
      } else {
        const unsigned hv = fv;
        asm volatile("" :: "s"(meta), "v"(hv) : "memory");
        const u32x3 w2 = __builtin_amdgcn_raw_buffer_load_b96(drs, lane * 12 + 2048, so, 17);
        const u32x4 w0 = __builtin_amdgcn_raw_buffer_load_b128(drs, lane * 16, so, 17);
        const u32x4 w1 = __builtin_amdgcn_raw_buffer_load_b128(drs, lane * 16 + 1024, so, 17);
        const unsigned fnext = __builtin_amdgcn_raw_buffer_load_b32(frs, foff, ((j + 1) & 7) * (int)(2 * HDR), 17);
        tv = __hip_atomic_load(tmo, __ATOMIC_RELAXED, __HIP_MEMORY_SCOPE_AGENT);
        const unsigned B = meta & 0x7Fu, nrec = (meta >> 16) & 63u;
#pragma unroll
        for (int e = 0; e < 2; ++e) {
          const unsigned rec = e ? hv >> 16 : hv & 0xFFFFu;
          const unsigned slot = (rec >> 6) & 31u;
          const unsigned d = ((rec >> 11) - 8u) * 4u;
          if (rec0 + e < nrec)
            __hip_atomic_fetch_add((LDS_AS unsigned*)(cor + (rec & 63u) * 32 + (slot >> 2) * 4), d << (8 * (slot & 3u)), __ATOMIC_RELAXED, __HIP_MEMORY_SCOPE_WORKGROUP);
        }
        asm volatile("" ::: "memory");
        const u32x4 c0 = *(LDS_AS u32x4*)(cor + lane * 32), c1 = *(LDS_AS u32x4*)(cor + lane * 32 + 16);
        asm volatile("" ::: "memory");
        *(LDS_AS u32x4*)(cor + lane * 32) = u32x4{0u, 0u, 0u, 0u};
        *(LDS_AS u32x4*)(cor + lane * 32 + 16) = u32x4{0u, 0u, 0u, 0u};
        if (j == 7) {
          pst = __builtin_amdgcn_readfirstlane(*(volatile LDS_AS unsigned*)(sync + 1 + (wave >> 1)));
          asm volatile("" ::: "memory");
          if (pst & 1u) { pa0 = other[0]; pa1 = other[1]; }
        }
        unsigned H[8];
        decode_hi(w2, B * 0x01010101u, H);
#pragma unroll
        for (int p = 0; p < 4; ++p) { H[p] -= c0[p]; H[4 + p] -= c1[p]; }
        const unsigned o80 = (meta & 0x80u) ? 0x00800080u : 0u;
#pragma unroll
        for (int k = 0; k < 4; ++k) mma1(wf[k][0], wf[k][1], expand(H, w0, w1, k, o80));
        fv = fnext;
      }
    }
  }
  if (tv == call) bad = true;
  if (!(pst & 1u)) {
    const unsigned mine = (unsigned)(size_t)((LDS_AS char*)lds + LDS_COR + wave * 2048 + lane * 32);
    const unsigned oth = (unsigned)(size_t)other, sta = (unsigned)(size_t)(sync + 1 + (wave >> 1)), inc = bad ? 3u : 1u;
    unsigned old; unsigned long long keep;
    asm volatile("s_nop 7\n\ts_nop 3\n\t"
                 "ds_write_b128 %4, %5\n\tds_write_b128 %4, %6 offset:16\n\t"
                 "s_mov_b64 %3, exec\n\ts_mov_b64 exec, 1\n\tds_add_rtn_u32 %0, %7, %8\n\ts_mov_b64 exec, %3\n\t"
                 "ds_read_b128 %1, %9\n\tds_read_b128 %2, %9 offset:16\n\ts_waitcnt lgkmcnt(0)"
                 : "=&v"(old), "=&v"(pa0), "=&v"(pa1), "=&s"(keep)
                 : "v"(mine), "v"(acc0), "v"(acc1), "v"(sta), "v"(inc), "v"(oth) : "memory");
    pst = __builtin_amdgcn_readfirstlane(old);
  }
  if (pst & 1u) {
    if (pst & 2u) bad = true;
    const f32x4 y0 = acc0 + pa0, y1 = acc1 + pa1;
    u32x4 ov = {cvt2(y0[0], y0[1]), cvt2(y0[2], y0[3]), cvt2(y1[0], y1[1]), cvt2(y1[2], y1[3])};
    if (bad) ov = u32x4{0x7FC07FC0u, 0x7FC07FC0u, 0x7FC07FC0u, 0x7FC07FC0u};
    asm volatile("global_store_dwordx4 %0, %1, off sc0 sc1\n\ts_nop 1" :: "v"(op), "v"(ov) : "memory");
  }
}
}

extern "C" void kernel_launch(int rank, int nranks, ncclComm_t comm, hipStream_t stream,
                              void* const* d_in, const int* in_sizes, int n_in,
                              void* d_out, int out_size, const int* in_dtypes, int out_dtype,
                              void* const* d_out_all, void* const* d_ws_all, size_t ws_bytes) {
  (void)comm; (void)in_sizes; (void)n_in; (void)out_size; (void)in_dtypes; (void)out_dtype; (void)d_out_all;
  if (nranks != NR || ws_bytes < WS_NEED || rank < 0 || rank >= NR) { fprintf(stderr, "bad config\n"); return; }
  static bool init = false;
  if (!init) {
    hipError_t e = hipFuncSetAttribute((const void*)ag_gemm, hipFuncAttributeMaxDynamicSharedMemorySize, LDS_BYTES);
    if (e != hipSuccess) { fprintf(stderr, "hipFuncSetAttribute: %s\n", hipGetErrorString(e)); return; }
    init = true;
  }
  Params P;
  P.x = (const char*)d_in[0]; P.w = (const char*)d_in[1]; P.out = (char*)d_out; P.myws = (char*)d_ws_all[rank];
  for (int q = 0; q < 14; ++q) P.peer[q] = (char*)d_ws_all[(rank + 1 + q % 7) % NR];
  P.rank = rank; P.pad = 0;
  hipLaunchKernelGGL(ag_gemm, dim3(256), dim3(THREADS), LDS_BYTES, stream, P);
}
```
